```python
import numpy as np

kernel_name = "gpu_gemm_ar_m4096_k4096_n16384_e4m3e5m2_dequant"


ABSMAX_RTOL = 0.05

COMPUTE_DTYPE = "float32"
EPILOGUE = "dequant"
IN_QUANT = None
OUT_QUANT = None
IN_FP8 = "e4m3e5m2"
IN_DTYPES = {"x": "e4m3", "w_mat": "e5m2",
             "scale_x": "float32", "scale_w": "float32"}
WORLD_SIZE = 8

def _snap_fp8(x, mant_bits, max_abs, min_normal):
    a = np.abs(x.astype(np.float64))
    _m, _e = np.frexp(a)
    q = np.where(a >= min_normal, np.ldexp(1.0, _e - 1 - mant_bits),
                 min_normal * 2.0 ** -mant_bits)
    snapped = np.minimum(np.round(a / q) * q, max_abs)
    return (np.sign(x) * snapped).astype(np.float32)

def setup_inputs():

    x = _snap_fp8(np.random.randn(4096, 4096).astype(np.float32), 3, 448.0, 2.0 ** -6)
    w_mat = _snap_fp8(np.random.randn(4096, 16384).astype(np.float32), 2, 57344.0, 2.0 ** -14)
    scale_x = (0.02 * (1.0 + np.random.rand(1))).astype(np.float32)
    scale_w = (0.02 * (1.0 + np.random.rand(1))).astype(np.float32)
    return {"x": x, "w_mat": w_mat, "scale_x": scale_x, "scale_w": scale_w}

def reference(x, w_mat, scale_x, scale_w):
    acc = (x @ w_mat).astype(np.float32)
    y = (acc * (scale_x * scale_w)).astype(np.float32)
    return y
```

```cpp
#include <hip/hip_runtime.h>
#include <rccl/rccl.h>
#include <cstdio>
#include <utility>

#define HIP_CHECK(expr) do { hipError_t _e = (expr); if (_e != hipSuccess) { \
    fprintf(stderr, "HIP error %s at %s:%d\n", hipGetErrorString(_e), __FILE__, __LINE__); \
    return; } } while (0)

#define DI __device__ __forceinline__
#define LAS __attribute__((address_space(3)))
#define GAP() __builtin_amdgcn_sched_barrier(0)
typedef int i32x4 __attribute__((ext_vector_type(4)));
typedef int i32x8 __attribute__((ext_vector_type(8)));
typedef float f32x4 __attribute__((ext_vector_type(4)));
typedef __amdgpu_buffer_rsrc_t rsrc_t;
#define AC10(p) "a" #p "0", "a" #p "1", "a" #p "2", "a" #p "3", "a" #p "4", "a" #p "5", "a" #p "6", "a" #p "7", "a" #p "8", "a" #p "9"
#define ACLOB "a0", "a1", "a2", "a3", "a4", "a5", "a6", "a7", "a8", "a9", AC10(1), AC10(2), AC10(3), AC10(4), AC10(5), AC10(6), AC10(7), AC10(8), AC10(9), \
  AC10(10), AC10(11), AC10(12), AC10(13), AC10(14), AC10(15), AC10(16), AC10(17), AC10(18), AC10(19), AC10(20), AC10(21), AC10(22), AC10(23), AC10(24), \
  "a250", "a251", "a252", "a253", "a254", "a255"
template <int... I> using iseq = std::integer_sequence<int, I...>;
template <int N> using mkseq = std::make_integer_sequence<int, N>;

constexpr int NR = 8, MM = 4096, NN = 16384, KS = 512;
constexpr unsigned WS_CNT = 0;
constexpr unsigned WS_TMO = 32u << 10;
constexpr unsigned WS_FLG = 64u << 10;
constexpr unsigned WS_DBG = 256u << 10;
constexpr unsigned WS_STG = 1u << 20;
constexpr unsigned STG_BYTES = 80u << 20, WB = 16u << 20;
constexpr unsigned WS_USED = WS_STG + 2u * STG_BYTES;
constexpr unsigned L_OUT = 128u << 10, L_RDY = L_OUT + 4096u, L_BAD = L_RDY + 4u, LDS_BYTES = L_RDY + 64u;
constexpr unsigned long long DEADLINE = 50000000ull;
constexpr int NU = 40;
#define PUSH_T 4

struct Params { const unsigned char* x; const unsigned char* w; const float* sx; const float* sw; float* out; char* ws[NR]; int rank; };

DI int jmap(int q, int rho) { return 4 * q + ((rho + q) & 3); }
DI unsigned rfl(unsigned v) { return __builtin_amdgcn_readfirstlane(v); }
DI unsigned ld_sys(const void* p) { return __hip_atomic_load((const unsigned*)p, __ATOMIC_RELAXED, __HIP_MEMORY_SCOPE_SYSTEM); }
DI void st_sys(void* p, unsigned v) { __hip_atomic_store((unsigned*)p, v, __ATOMIC_RELAXED, __HIP_MEMORY_SCOPE_SYSTEM); }
DI unsigned vmcnt_now() { const unsigned r = __builtin_amdgcn_s_getreg(7 | (31 << 11)); return (r & 15u) | ((r >> 18) & 48u); }
DI unsigned flag_off(unsigned uoff) { return WS_FLG + (uoff >> 17) * 128u + (((uoff >> 12) & 1u) * 2u + ((uoff >> 14) & 1u)) * 4u; }

struct Frag4 { i32x4 lo[4], hi[4]; };

struct Ctx {
  unsigned aoff, boff, aoffH, boffH, voff, ldsb;
  rsrc_t rs;
  unsigned sA, m0base, left;
  unsigned call, stg; int lane;
  char* wsme;
  unsigned pw, px, fv; bool ok;
  rsrc_t dst[7]; unsigned tbl, b, nb, uf, offS, offD, v; bool go, fl; char* fbase; i32x4 cur;
};
enum { R_LOAD = 0, R_POLL = 1, R_PUSH = 2 };

template <int SLOT, bool ISB, int I> DI void rd1(Ctx& c, Frag4& f) {
  constexpr int off = (SLOT & 1) * 32768 + (ISB ? 16384 : 0) + (I & 1) * 8192 + (I >> 1) * 1024;
  const unsigned base = SLOT >= 2 ? (ISB ? c.boffH : c.aoffH) : (ISB ? c.boff : c.aoff);
  const i32x4 v = *(const LAS i32x4*)((LAS char*)(size_t)base + off);
  if constexpr (I & 1) f.hi[I >> 1] = v; else f.lo[I >> 1] = v;
}
DI i32x8 cat(i32x4 a, i32x4 b) { return __builtin_shufflevector(a, b, 0, 1, 2, 3, 4, 5, 6, 7); }
template <bool ZERO, int AH, int BH, int G> DI void mma1(const Frag4& a, const Frag4& b) {
  constexpr int i = G >> 2, j = G & 3, r = 4 * (8 * (AH * 4 + i) + BH * 4 + j);
  if constexpr (ZERO)
    asm volatile("v_mfma_f32_16x16x128_f8f6f4 a[%c2:%c3], %0, %1, 0 blgp:1" :: "v"(cat(a.lo[i], a.hi[i])), "v"(cat(b.lo[j], b.hi[j])), "i"(r), "i"(r + 3) : ACLOB);
  else
    asm volatile("v_mfma_f32_16x16x128_f8f6f4 a[%c2:%c3], %0, %1, a[%c2:%c3] blgp:1" :: "v"(cat(a.lo[i], a.hi[i])), "v"(cat(b.lo[j], b.hi[j])), "i"(r), "i"(r + 3) : ACLOB);
}
template <int R> DI float acc_rd() { float v; asm volatile("v_accvgpr_read_b32 %0, a%c1" : "=v"(v) : "i"(R)); return v; }
template <int R> DI void acc_zero1() { asm volatile("v_accvgpr_write_b32 a%c0, 0" :: "i"(R) : ACLOB); }
template <int... R> DI void acc_zero(iseq<R...>) { (acc_zero1<R>(), ...); }
template <int S> DI void out1(float* o, float sc) {
  constexpr int i = S >> 3, r = (S >> 1) & 3, h = S & 1, a = 4 * (8 * i + 4 * h) + r;
  const f32x4 v = {acc_rd<a>(), acc_rd<a + 4>(), acc_rd<a + 8>(), acc_rd<a + 12>()};
  __builtin_nontemporal_store(v * sc, (f32x4*)(o + (size_t)(16 * i + r) * NN + 64 * h));
}
template <int... S> DI void out_all(float* o, float sc, iseq<S...>) { (out1<S>(o, sc), ...); }
template <int SLOT, int P> DI void dma1(Ctx& c) {
  constexpr int g = P >> 2, q = P & 3;
  const unsigned so = c.sA + g * 4096;
  if constexpr (q == 0)
    asm volatile("s_add_u32 m0, %0, %1\n\ts_nop 0\n\tbuffer_load_dwordx4 %2, %3, %4 offen sc0 sc1 lds" :: "s"(c.m0base), "i"(SLOT * 32768 + g * 4096), "v"(c.voff), "s"(c.rs), "s"(so) : "memory", "scc");
  else
    asm volatile("buffer_load_dwordx4 %0, %1, %2 offen offset:%3 sc0 sc1 lds" :: "v"(c.voff), "s"(c.rs), "s"(so), "i"(q * 1024) : "memory");
}
template <int SLOT, int... P> DI void dma_all(Ctx& c, iseq<P...>) { (dma1<SLOT, P>(c), ...); }
template <int SLOT, bool ISB, int... I> DI void rd_all(Ctx& c, Frag4& f, iseq<I...>) { (rd1<SLOT, ISB, I>(c, f), ...); }

DI void st_wt(const i32x4& d, unsigned voff, const rsrc_t& r, unsigned soff) {
  asm volatile("buffer_store_dwordx4 %0, %1, %2, %3 offen sc0 sc1\n\ts_nop 1" :: "v"(d), "v"(voff), "s"(r), "s"(soff) : "memory");
}
DI void push_fetch(Ctx& c) { c.cur = *(const LAS i32x4*)(size_t)(c.ldsb + L_OUT + ((c.b & 3u) << 10) + c.voff); }
DI void push_offS(Ctx& c) { c.offS = c.stg + (unsigned)__builtin_amdgcn_readlane((int)c.tbl, (int)(c.b >> 2)) + ((c.b & 3u) << 10); asm volatile("" : "+s"(c.offS)); }
DI void push_offD(Ctx& c) { c.offD = c.stg + (unsigned)__builtin_amdgcn_readlane((int)c.tbl, (int)((c.b >> 2) + 1u)) + ((c.b & 3u) << 10); asm volatile("" : "+s"(c.offD)); }
DI void outbox_dma(Ctx& c, unsigned slot, unsigned soff) {
  const unsigned m0v = c.ldsb + L_OUT + (slot << 10);
  asm volatile("s_mov_b32 m0, %0\n\ts_nop 0\n\tbuffer_load_dwordx4 %1, %2, %3 offen sc0 sc1 lds" :: "s"(m0v), "v"(c.voff), "s"(c.rs), "s"(soff) : "memory");
}
DI void push_decide(Ctx& c) {
  c.go = (c.b < c.nb) & (c.v <= (unsigned)PUSH_T);
  c.fl = (int)(c.b - ((c.v + 7u) >> 3)) >= (int)(4u * c.uf + 4u);
}
DI void push_flag(Ctx& c) {
  const unsigned fo = flag_off((unsigned)__builtin_amdgcn_readlane((int)c.tbl, (int)c.uf));
  if (c.lane < NR) st_sys(c.fbase + fo, c.call);
  c.uf += 1; c.fl = false;
}
DI void push_plain(Ctx& c) {
  push_fetch(c); push_offS(c); push_offD(c);
  asm volatile("s_nop 4" ::: "memory");
#pragma unroll
  for (int i = 0; i < 7; ++i) st_wt(c.cur, c.voff, c.dst[i], c.offS);
  outbox_dma(c, c.b & 3u, c.offD);
  c.b += 1; c.go = false;
}
DI void push_spin_step(Ctx& c) {
  c.v = vmcnt_now(); push_decide(c);
  if (c.fl) push_flag(c);
  if (c.go) push_plain(c); else __builtin_amdgcn_s_sleep(1);
}

template <int ROLE, int VAR, int AH, int BH, int RSLOT, bool RISB, int DSLOT, int DP0, int G>
DI void gap1(Ctx& c, const Frag4& a, const Frag4& b, Frag4& dst) {
  mma1<false, AH, BH, G>(a, b);
  if constexpr (G < 8) rd1<RSLOT, RISB, G>(c, dst);
  if constexpr (ROLE == R_LOAD && G >= 8) dma1<DSLOT, DP0 + (G - 8)>(c);
  if constexpr (ROLE == R_POLL && VAR == 1 && G == 8) c.fv = ld_sys(c.wsme + (c.lane < 32 ? c.pw : c.px) + (c.lane & 31) * 4);
  if constexpr (ROLE == R_POLL && VAR == 2 && G == 8) c.ok = __all((int)(c.fv - c.call) >= 0);
  if constexpr (ROLE == R_PUSH && VAR == 1) {
    if constexpr (G == 0) push_fetch(c);
    if constexpr (G == 2) push_offS(c);
    if constexpr (G == 4) push_offD(c);
    if constexpr (G >= 8 && G < 15) st_wt(c.cur, c.voff, c.dst[G - 8], c.offS);
    if constexpr (G == 15) { outbox_dma(c, c.b & 3u, c.offD); c.b += 1; c.go = false; c.fl = false; }
  }
  if constexpr (ROLE == R_PUSH && VAR == 0) {
    if constexpr (G == 11) { c.v = vmcnt_now(); asm volatile("" : "+s"(c.v)); }
    if constexpr (G == 13) push_decide(c);
  }
  GAP();
}
template <int ROLE, int VAR, int AH, int BH, int RSLOT, bool RISB, int DSLOT, int DP0, int... G>
DI void quad_(Ctx& c, const Frag4& a, const Frag4& b, Frag4& dst, iseq<G...>) { (gap1<ROLE, VAR, AH, BH, RSLOT, RISB, DSLOT, DP0, G>(c, a, b, dst), ...); }
template <int ROLE, int PV, int AH, int BH, int RSLOT, bool RISB, int DSLOT, int DP0>
DI void quad(Ctx& c, const Frag4& a, const Frag4& b, Frag4& dst) {
  if constexpr (ROLE == R_PUSH) {
    if (c.go) quad_<ROLE, 1, AH, BH, RSLOT, RISB, DSLOT, DP0>(c, a, b, dst, mkseq<16>{});
    else {
      quad_<ROLE, 0, AH, BH, RSLOT, RISB, DSLOT, DP0>(c, a, b, dst, mkseq<16>{});
      if (__builtin_expect(c.fl, 0)) push_flag(c);
    }
  } else quad_<ROLE, PV, AH, BH, RSLOT, RISB, DSLOT, DP0>(c, a, b, dst, mkseq<16>{});
}
template <int ROLE> DI void epoch_end(Ctx& c, unsigned nxt) {
  if constexpr (ROLE == R_LOAD) {
    c.sA += 16384; if (--c.left == 0) { c.sA = nxt; c.left = 64; }
    asm volatile("s_waitcnt vmcnt(32) lgkmcnt(0)\n\ts_barrier" ::: "memory");
  } else asm volatile("s_waitcnt lgkmcnt(0)\n\ts_barrier" ::: "memory");
  GAP();
}
DI void lds_st(Ctx& c, unsigned off, unsigned v) { *(volatile LAS unsigned*)(size_t)(c.ldsb + off) = v; }
DI unsigned lds_ld(Ctx& c, unsigned off) { return *(volatile LAS unsigned*)(size_t)(c.ldsb + off); }

DI void poll_lines(Ctx& c, int pass, int ti, int q) {
  const int rho = pass >> 3, ks = pass & 7;
  c.pw = WS_FLG + (unsigned)(128 + jmap(q, rho) * 8 + ks) * 128u;
  c.px = rho == 0 ? WS_FLG + (unsigned)(ti * 8 + ks) * 128u : c.pw;
}
DI bool wait_flags(Ctx& c, const char* addr, bool active) {
  const unsigned long long t0 = __builtin_amdgcn_s_memrealtime();
  for (;;) {
    const unsigned v = active ? ld_sys(addr) : c.call;
    if (__all((int)(v - c.call) >= 0)) return true;
    if (rfl(__hip_atomic_load((const unsigned*)(c.wsme + WS_TMO), __ATOMIC_RELAXED, __HIP_MEMORY_SCOPE_AGENT)) == c.call ||
        __builtin_amdgcn_s_memrealtime() - t0 > DEADLINE) {
      if (c.lane == 0) { __hip_atomic_store((unsigned*)(c.wsme + WS_TMO), c.call, __ATOMIC_RELAXED, __HIP_MEMORY_SCOPE_AGENT); lds_st(c, L_BAD, 1u); }
      return false;
    }
    __builtin_amdgcn_s_sleep(2);
  }
}
template <int ROLE> DI void gate(Ctx& c, int pass, int ti, int q) {
  if constexpr (ROLE == R_POLL) {
    if (!c.ok) { wait_flags(c, c.wsme + (c.lane < 32 ? c.pw : c.px) + (c.lane & 31) * 4, true); if (c.lane == 0) lds_st(c, L_RDY, (unsigned)pass + 1u); }
    c.ok = false;
    poll_lines(c, pass < 31 ? pass + 1 : 31, ti, q);
  }
  if constexpr (ROLE == R_PUSH) {
    while (c.uf < c.nb / 4u && rfl(lds_ld(c, L_RDY)) <= (unsigned)pass) push_spin_step(c);
  }
}

template <int ROLE>
DI void body(Ctx& c, float* __restrict__ out, float scale, int ti, int q, int wr, int wc) {
  Frag4 A0, A1, BX, BY;
  auto src = [&](int rho) { return c.stg + (wc == 0 ? (unsigned)ti << 20 : WB + ((unsigned)jmap(q, rho) << 20)); };
  c.left = 64; c.sA = src(0);
  if constexpr (ROLE == R_POLL) { c.ok = false; poll_lines(c, 0, ti, q); }
  gate<ROLE>(c, 0, ti, q);
  asm volatile("s_waitcnt lgkmcnt(0)\n\ts_barrier" ::: "memory");
  if constexpr (ROLE == R_LOAD) {
    dma_all<0>(c, mkseq<16>{}); c.sA += 16384;
    dma_all<1>(c, mkseq<16>{}); c.sA += 16384;
    dma_all<2>(c, mkseq<16>{}); c.left -= 2;
  }
  epoch_end<ROLE>(c, 0);
  rd_all<0, false>(c, A0, mkseq<8>{}); rd_all<0, true>(c, BX, mkseq<8>{});
  if constexpr (ROLE == R_LOAD) dma_all<3>(c, mkseq<16>{});
  epoch_end<ROLE>(c, 0);
#pragma unroll 1
  for (int rho = 0; rho < 4; ++rho) {
    const unsigned nxt = src(rho < 3 ? rho + 1 : 3);
    acc_zero(mkseq<256>{});
    asm volatile("s_nop 3" ::: "memory"); GAP();
#pragma unroll 1
    for (int it = 0; it < 16; ++it) {
      quad<ROLE, 1, 0, 0, 1, true, 0, 0>(c, A0, BX, BY);
      quad<ROLE, 0, 0, 1, 1, false, 0, 8>(c, A0, BY, A1);
      epoch_end<ROLE>(c, nxt);
      quad<ROLE, 0, 1, 1, 2, false, 1, 0>(c, A1, BY, A0);
      quad<ROLE, 0, 1, 0, 2, true, 1, 8>(c, A1, BX, BY);
      epoch_end<ROLE>(c, nxt);
      quad<ROLE, 2, 0, 0, 3, true, 2, 0>(c, A0, BY, BX);
      const int pass = rho * 8 + ((it + 2) >> 1);
      if constexpr (ROLE == R_POLL) { if (c.ok && c.lane == 0) lds_st(c, L_RDY, (unsigned)pass + 1u); }
      quad<ROLE, 0, 0, 1, 3, false, 2, 8>(c, A0, BX, A1);
      epoch_end<ROLE>(c, nxt);
      quad<ROLE, 0, 1, 1, 0, false, 3, 0>(c, A1, BX, A0);
      quad<ROLE, 0, 1, 0, 0, true, 3, 8>(c, A1, BY, BX);
      if ((it & 1) == 0 && pass < 32) gate<ROLE>(c, pass, ti, q);
      epoch_end<ROLE>(c, nxt);
    }
    asm volatile("s_nop 15" ::: "memory"); GAP();
    const float sc = lds_ld(c, L_BAD) ? __builtin_nanf("") : scale;
    float* o = out + (size_t)(ti * 256 + wr * 128 + 4 * (c.lane >> 4)) * NN + jmap(q, rho) * 256 + wc * 128 + 4 * (c.lane & 15);
    out_all(o, sc, mkseq<64>{});
    if constexpr (ROLE == R_PUSH) { c.go = false; c.fl = false; }
    GAP();
  }
  if constexpr (ROLE == R_PUSH) { while (c.uf < c.nb / 4u) push_spin_step(c); }
  asm volatile("s_waitcnt vmcnt(0)" ::: "memory");
}

DI void unit_load(const Params& p, unsigned uoff, int lane, unsigned (&d)[16]) {
  const unsigned ks = (uoff >> 17) & 7u, sub = ((uoff >> 12) & 1u) * 128u + ((uoff >> 14) & 1u) * 64u, g = lane >> 4, c = lane & 15;
  if (uoff < WB) {
    const unsigned char* s = p.x + (size_t)((uoff >> 20) * 256u + sub + c) * KS + ks * 64u + g * 16u;
#pragma unroll
    for (int f = 0; f < 4; ++f) { const i32x4 v = *(const i32x4*)(s + f * 16 * KS); d[4 * f] = v[0]; d[4 * f + 1] = v[1]; d[4 * f + 2] = v[2]; d[4 * f + 3] = v[3]; }
  } else {
    const unsigned char* s = p.w + (size_t)(ks * 64u + g * 16u) * NN + ((uoff - WB) >> 20) * 256u + sub + c * 4u;
#pragma unroll
    for (int i = 0; i < 16; ++i) d[i] = *(const unsigned*)(s + (size_t)i * NN);
  }
}
DI void unit_fix(unsigned uoff, const unsigned (&d)[16], i32x4 (&o)[4]) {
  if (uoff < WB) {
#pragma unroll
    for (int f = 0; f < 4; ++f) o[f] = (i32x4){(int)d[4 * f], (int)d[4 * f + 1], (int)d[4 * f + 2], (int)d[4 * f + 3]};
  } else {
#pragma unroll
    for (int a = 0; a < 4; ++a) {
      const unsigned t0 = __builtin_amdgcn_perm(d[4 * a + 1], d[4 * a], 0x05010400u), t1 = __builtin_amdgcn_perm(d[4 * a + 1], d[4 * a], 0x07030602u);
      const unsigned t2 = __builtin_amdgcn_perm(d[4 * a + 3], d[4 * a + 2], 0x05010400u), t3 = __builtin_amdgcn_perm(d[4 * a + 3], d[4 * a + 2], 0x07030602u);
      o[0][a] = (int)__builtin_amdgcn_perm(t2, t0, 0x05040100u); o[1][a] = (int)__builtin_amdgcn_perm(t2, t0, 0x07060302u);
      o[2][a] = (int)__builtin_amdgcn_perm(t3, t1, 0x05040100u); o[3][a] = (int)__builtin_amdgcn_perm(t3, t1, 0x07060302u);
    }
  }
}
DI rsrc_t mk_rsrc(char* base) { return __builtin_amdgcn_make_buffer_rsrc(base, 0, WS_USED, 0x00020000); }

__global__ void __launch_bounds__(256, 1) fused_gemm_gather(Params p) {
  extern __shared__ __attribute__((aligned(1024))) char smem[];
  const int tid = threadIdx.x, wid = __builtin_amdgcn_readfirstlane(tid >> 6), lane = tid & 63, wr = wid >> 1, wc = wid & 1;
  const int b = blockIdx.x, xc = b & 7, s = b >> 3;
  const int ti = (xc >> 1) * 4 + (s & 3), q = (xc & 1) * 8 + (s >> 2), t = xc >> 1;
  const unsigned rank = (unsigned)p.rank & 7u;
  Ctx c;
  c.lane = lane; c.wsme = p.ws[rank];
  const unsigned cnt = ld_sys(c.wsme + WS_CNT + b * 128);
  c.ldsb = (unsigned)(size_t)(LAS char*)smem;
  c.voff = lane * 16;
  c.aoff = c.ldsb + c.voff + wr * 4096; c.boff = c.ldsb + c.voff + wc * 4096; c.aoffH = c.aoff + 65536; c.boffH = c.boff + 65536;
  asm volatile("" : "+v"(c.aoff), "+v"(c.boff), "+v"(c.aoffH), "+v"(c.boffH));
  c.m0base = c.ldsb + wc * 16384;
  c.rs = mk_rsrc(c.wsme);
  if (tid == 0) { lds_st(c, L_RDY, 0u); lds_st(c, L_BAD, 0u); }
  {
    const int n = lane < NU ? lane : NU - 1;
    const bool isx = n < 16 && (n & 1) == 0;
    const int rho = n < 16 ? 0 : (n - 8) >> 3, ks = n < 16 ? n >> 1 : n & 7;
    c.tbl = (isx ? (unsigned)q << 20 : WB + ((unsigned)jmap(q, rho) << 20)) + (unsigned)ks * 131072u + (rank >> 1) * 32768u + (unsigned)(t & 1) * 16384u + (rank & 1u) * 8192u + (unsigned)(t >> 1) * 4096u;
  }
  const float scale = p.sx[0] * p.sw[0];
  const int hw = (s & 3) * 4 + wid;
  c.b = 0; c.nb = 0; c.uf = 0; c.go = false; c.fl = false;
  if (hw != 3) {
    const int idx = hw < 3 ? hw : hw - 1;
    unsigned d[3][16], uo[3];
#pragma unroll
    for (int i = 0; i < 3; ++i) { const int n = 2 + idx + 15 * i; uo[i] = (unsigned)__builtin_amdgcn_readlane((int)c.tbl, n < NU ? n : NU - 1); unit_load(p, uo[i], lane, d[i]); }
    c.call = rfl(cnt) + 1u; c.stg = WS_STG + (c.call & 1u) * STG_BYTES;
#pragma unroll
    for (int i = 0; i < 3; ++i) if (2 + idx + 15 * i < NU) {
      i32x4 o[4]; unit_fix(uo[i], d[i], o);
      asm volatile("s_nop 4" ::: "memory");
#pragma unroll
      for (int f = 0; f < 4; ++f) st_wt(o[f], c.voff, c.rs, c.stg + uo[i] + f * 1024);
    }
    asm volatile("s_waitcnt vmcnt(0)" ::: "memory");
    const int n = lane - 2 - idx;
    if (lane < NU && n >= 0 && n % 15 == 0) st_sys(c.wsme + flag_off(c.tbl) + rank * 16u, c.call);
  } else {
    unsigned d[2][16], uo[2];
#pragma unroll
    for (int i = 0; i < 2; ++i) { uo[i] = (unsigned)__builtin_amdgcn_readlane((int)c.tbl, i); unit_load(p, uo[i], lane, d[i]); }
    const unsigned pid = (unsigned)(q * 4 + t);
#pragma unroll
    for (int i = 0; i < 7; ++i) c.dst[i] = mk_rsrc(p.ws[(rank + 1u + (i + pid) % 7u) & 7u]);
    c.fbase = p.ws[lane & 7] + rank * 16u;
    c.call = rfl(cnt) + 1u; c.stg = WS_STG + (c.call & 1u) * STG_BYTES;
    c.nb = 4 * NU;
    i32x4 o[2][4];
    unit_fix(uo[0], d[0], o[0]); unit_fix(uo[1], d[1], o[1]);
#pragma unroll
    for (int u = 0; u < 2; ++u) {
      if (u == 1) {
        if (!wait_flags(c, c.wsme + flag_off(c.tbl) + rank * 16u, lane >= 2 && lane < NU)) c.nb = 0;
        const unsigned o2 = c.stg + (unsigned)__builtin_amdgcn_readlane((int)c.tbl, 2);
        asm volatile("s_nop 4" ::: "memory");
#pragma unroll
        for (int f = 0; f < 4; ++f) outbox_dma(c, f, o2 + f * 1024);
      }
#pragma unroll
      for (int f = 0; f < 4; ++f) {
        const unsigned so = c.stg + uo[u] + f * 1024;
        asm volatile("s_nop 4" ::: "memory");
#pragma unroll
        for (int i = 0; i < 7; ++i) st_wt(o[u][f], c.voff, c.dst[i], so);
        st_wt(o[u][f], c.voff, c.rs, so);
        c.b += 1;
        asm volatile("s_waitcnt vmcnt(%0)" :: "i"(PUSH_T) : "memory");
        c.v = vmcnt_now(); push_decide(c);
        if (c.fl) push_flag(c);
      }
    }
    c.go = false; c.fl = false;
  }
  asm volatile("s_waitcnt lgkmcnt(0)\n\ts_barrier" ::: "memory");
  if (wid < 2) body<R_LOAD>(c, p.out, scale, ti, q, wr, wc);
  else if (wid == 2) body<R_POLL>(c, p.out, scale, ti, q, wr, wc);
  else body<R_PUSH>(c, p.out, scale, ti, q, wr, wc);
  if (tid == 0) st_sys(c.wsme + WS_CNT + b * 128, c.call);
}

extern "C" void kernel_launch(int rank, int nranks, ncclComm_t comm, hipStream_t stream,
                              void* const* d_in, const int* in_sizes, int n_in,
                              void* d_out, int out_size, const int* in_dtypes, int out_dtype,
                              void* const* d_out_all, void* const* d_ws_all, size_t ws_bytes) {
  (void)comm; (void)in_dtypes; (void)out_dtype; (void)d_out_all;
  if (nranks != NR || n_in != 4 || (unsigned)rank >= (unsigned)NR || ws_bytes < (size_t)WS_USED || in_sizes[0] != MM * KS || in_sizes[1] != KS * NN || out_size != MM * NN) {
    fprintf(stderr, "kernel_launch: unexpected problem shape\n"); return;
  }
  HIP_CHECK(hipFuncSetAttribute((const void*)fused_gemm_gather, hipFuncAttributeMaxDynamicSharedMemorySize, (int)LDS_BYTES));
  Params p;
  p.x = (const unsigned char*)d_in[0]; p.w = (const unsigned char*)d_in[1]; p.sx = (const float*)d_in[2]; p.sw = (const float*)d_in[3];
  p.out = (float*)d_out; p.rank = rank;
  for (int i = 0; i < NR; ++i) p.ws[i] = (char*)d_ws_all[i];
  fused_gemm_gather<<<dim3(256), dim3(256), LDS_BYTES, stream>>>(p);
  HIP_CHECK(hipGetLastError());
}
```

```cpp
#include <hip/hip_runtime.h>
#include <rccl/rccl.h>
#include <cstdio>
#include <utility>
#include <algorithm>

#define HIP_CHECK(expr) do { hipError_t _e = (expr); if (_e != hipSuccess) { \
    fprintf(stderr, "HIP error %s at %s:%d\n", hipGetErrorString(_e), __FILE__, __LINE__); \
    return; } } while (0)

#define DI __device__ __forceinline__
#define HD __host__ __device__ __forceinline__
#define LAS __attribute__((address_space(3)))
#define GAP() __builtin_amdgcn_sched_barrier(0)
typedef int i32x4 __attribute__((ext_vector_type(4)));
typedef int i32x8 __attribute__((ext_vector_type(8)));
typedef float f32x4 __attribute__((ext_vector_type(4)));
typedef __amdgpu_buffer_rsrc_t rsrc_t;
#define AC10(p) "a" #p "0", "a" #p "1", "a" #p "2", "a" #p "3", "a" #p "4", "a" #p "5", "a" #p "6", "a" #p "7", "a" #p "8", "a" #p "9"
#define ACLOB "a0", "a1", "a2", "a3", "a4", "a5", "a6", "a7", "a8", "a9", AC10(1), AC10(2), AC10(3), AC10(4), AC10(5), AC10(6), AC10(7), AC10(8), AC10(9), \
  AC10(10), AC10(11), AC10(12), AC10(13), AC10(14), AC10(15), AC10(16), AC10(17), AC10(18), AC10(19), AC10(20), AC10(21), AC10(22), AC10(23), AC10(24), \
  "a250", "a251", "a252", "a253", "a254", "a255"
template <int... I> using iseq = std::integer_sequence<int, I...>;
template <int N> using mkseq = std::make_integer_sequence<int, N>;

constexpr int NR = 8, MM = 4096, NN = 16384, KS = 512;
constexpr unsigned WS_CNT = 0;
constexpr unsigned WS_TMO = 32u << 10;
constexpr unsigned WS_TKT = WS_TMO + 128u;
constexpr unsigned WS_FLG = 64u << 10;
constexpr unsigned WS_DBG = 256u << 10;
constexpr unsigned WS_STG = 1u << 20;
constexpr unsigned STG_BYTES = 80u << 20, WB = 16u << 20;
constexpr unsigned WS_USED = WS_STG + 2u * STG_BYTES;
constexpr int NITEM = 640;
constexpr unsigned L_RDY = 128u << 10, L_BAD = L_RDY + 4u, L_FULL = L_RDY + 16u, L_FREE = L_RDY + 32u, LDS_BYTES = L_RDY + 64u;
constexpr unsigned long long DEADLINE = 50000000ull;
#define RING 2
constexpr unsigned NSEND = 32, NUNIT = NITEM * 4, L_ORD = 64u << 10, L_HDR = 68u << 10;
#define PUSH_W 7
#define SKEW 1.5
#define XLEAD 0.5
#define PH5 1.0
#define NP4 0
#define NP5 0
#define PST 0

struct Params { const unsigned char* x; const unsigned char* w; const float* sx; const float* sw; float* out; char* ws[NR]; int rank; int pad; unsigned short order[NITEM]; };

HD int ntile(int q) { return q < 4 ? 5 : 4; }
HD int jmap(int q, int r) { if (q < 4) { if (r == 0) return 60 + q; --r; } return 4 * q + ((r + q) & 3); }
DI unsigned rfl(unsigned v) { return __builtin_amdgcn_readfirstlane(v); }
DI unsigned ld_sys(const void* p) { return __hip_atomic_load((const unsigned*)p, __ATOMIC_RELAXED, __HIP_MEMORY_SCOPE_SYSTEM); }
DI void st_sys(void* p, unsigned v) { __hip_atomic_store((unsigned*)p, v, __ATOMIC_RELAXED, __HIP_MEMORY_SCOPE_SYSTEM); }
DI unsigned unit_off(unsigned strip, unsigned pos, unsigned t) { return (strip << 20) + (pos >> 1) * 32768u + (t & 1u) * 16384u + (pos & 1u) * 8192u + (t >> 1) * 4096u; }

struct Frag4 { i32x4 lo[4], hi[4]; };

struct Ctx {
  unsigned aoff, boff, aoffH, boffH, voff, ldsb;
  rsrc_t rs;
  unsigned sA, m0base, left;
  unsigned call, stg; int lane;
  char* wsme;
  unsigned pa, fv, nleft, rank; bool ok;
};
enum { R_LOAD = 0, R_POLL = 1, R_NONE = 2 };
#define STAMP(i) do { } while (0)

template <int SLOT, bool ISB, int I> DI void rd1(Ctx& c, Frag4& f) {
  constexpr int off = (SLOT & 1) * 32768 + (ISB ? 16384 : 0) + (I & 1) * 8192 + (I >> 1) * 1024;
  const unsigned base = SLOT >= 2 ? (ISB ? c.boffH : c.aoffH) : (ISB ? c.boff : c.aoff);
  const i32x4 v = *(const LAS i32x4*)((LAS char*)(size_t)base + off);
  if constexpr (I & 1) f.hi[I >> 1] = v; else f.lo[I >> 1] = v;
}
DI i32x8 cat(i32x4 a, i32x4 b) { return __builtin_shufflevector(a, b, 0, 1, 2, 3, 4, 5, 6, 7); }
template <bool ZERO, int AH, int BH, int G> DI void mma1(const Frag4& a, const Frag4& b) {
  constexpr int i = G >> 2, j = G & 3, r = 4 * (8 * (AH * 4 + i) + BH * 4 + j);
  if constexpr (ZERO)
    asm volatile("v_mfma_f32_16x16x128_f8f6f4 a[%c2:%c3], %0, %1, 0 blgp:1" :: "v"(cat(a.lo[i], a.hi[i])), "v"(cat(b.lo[j], b.hi[j])), "i"(r), "i"(r + 3) : ACLOB);
  else
    asm volatile("v_mfma_f32_16x16x128_f8f6f4 a[%c2:%c3], %0, %1, a[%c2:%c3] blgp:1" :: "v"(cat(a.lo[i], a.hi[i])), "v"(cat(b.lo[j], b.hi[j])), "i"(r), "i"(r + 3) : ACLOB);
}
template <int R> DI float acc_rd() { float v; asm volatile("v_accvgpr_read_b32 %0, a%c1" : "=v"(v) : "i"(R)); return v; }
template <int S> DI f32x4 acc4() {
  constexpr int i = S >> 3, r = (S >> 1) & 3, h = S & 1, a = 4 * (8 * i + 4 * h) + r;
  return (f32x4){acc_rd<a>(), acc_rd<a + 4>(), acc_rd<a + 8>(), acc_rd<a + 12>()};
}
template <int S> DI f32x4* optr(float* o) { return (f32x4*)(o + (size_t)(16 * (S >> 3) + ((S >> 1) & 3)) * NN + 64 * (S & 1)); }
template <bool PART, int S> DI void out1(float* o, float sc) {
  if constexpr (PART && PST == 0) *optr<S>(o) = acc4<S>() * sc;
  else __builtin_nontemporal_store(acc4<S>() * sc, optr<S>(o));
}
template <bool PART, int... S> DI void out_all(float* o, float sc, iseq<S...>) { (out1<PART, S>(o, sc), ...); }
template <int S0, int... S> DI void old_ld(float* o, f32x4 (&v)[16], iseq<S...>) { ((v[S] = __builtin_nontemporal_load(optr<S0 + S>(o))), ...); }
template <int S0, int... S> DI void add_st(float* o, float sc, const f32x4 (&v)[16], iseq<S...>) { (__builtin_nontemporal_store(acc4<S0 + S>() * sc + v[S], optr<S0 + S>(o)), ...); }
DI void out_add(float* o, float sc) {
  f32x4 u[16], v[16];
  old_ld<0>(o, u, mkseq<16>{}); old_ld<16>(o, v, mkseq<16>{}); GAP();
  add_st<0>(o, sc, u, mkseq<16>{}); GAP();
  old_ld<32>(o, u, mkseq<16>{}); GAP();
  add_st<16>(o, sc, v, mkseq<16>{}); GAP();
  old_ld<48>(o, v, mkseq<16>{}); GAP();
  add_st<32>(o, sc, u, mkseq<16>{}); GAP();
  add_st<48>(o, sc, v, mkseq<16>{});
}
template <int SLOT, int P> DI void dma1(Ctx& c) {
  constexpr int g = P >> 2, q = P & 3;
  const unsigned so = c.sA + g * 4096;
  if constexpr (q == 0)
    asm volatile("s_add_u32 m0, %0, %1\n\ts_nop 0\n\tbuffer_load_dwordx4 %2, %3, %4 offen sc0 sc1 lds" :: "s"(c.m0base), "i"(SLOT * 32768 + g * 4096), "v"(c.voff), "s"(c.rs), "s"(so) : "memory", "scc");
  else
    asm volatile("buffer_load_dwordx4 %0, %1, %2 offen offset:%3 sc0 sc1 lds" :: "v"(c.voff), "s"(c.rs), "s"(so), "i"(q * 1024) : "memory");
}
template <int SLOT, int... P> DI void dma_all(Ctx& c, iseq<P...>) { (dma1<SLOT, P>(c), ...); }
template <int SLOT, bool ISB, int... I> DI void rd_all(Ctx& c, Frag4& f, iseq<I...>) { (rd1<SLOT, ISB, I>(c, f), ...); }
DI void st_wt(const i32x4& d, unsigned voff, const rsrc_t& r, unsigned soff) {
  asm volatile("buffer_store_dwordx4 %0, %1, %2, %3 offen sc0 sc1\n\ts_nop 1" :: "v"(d), "v"(voff), "s"(r), "s"(soff) : "memory");
}

template <int ROLE, int VAR, bool Z, int AH, int BH, int RSLOT, bool RISB, int DSLOT, int DP0, int G>
DI void gap1(Ctx& c, const Frag4& a, const Frag4& b, Frag4& dst) {
  mma1<Z, AH, BH, G>(a, b);
  if constexpr (G < 8) rd1<RSLOT, RISB, G>(c, dst);
  if constexpr (ROLE == R_LOAD && G >= 8) dma1<DSLOT, DP0 + (G - 8)>(c);
  if constexpr (ROLE == R_POLL && VAR == 1 && G == 8) c.fv = ld_sys(c.wsme + c.pa);
  if constexpr (ROLE == R_POLL && VAR == 2 && G == 10) c.ok = __all((int)(c.fv - c.call) >= 0);
  GAP();
}
template <int ROLE, int VAR, bool Z, int AH, int BH, int RSLOT, bool RISB, int DSLOT, int DP0, int... G>
DI void quad_(Ctx& c, const Frag4& a, const Frag4& b, Frag4& dst, iseq<G...>) { (gap1<ROLE, VAR, Z, AH, BH, RSLOT, RISB, DSLOT, DP0, G>(c, a, b, dst), ...); }
template <int ROLE, int VAR, bool Z, int AH, int BH, int RSLOT, bool RISB, int DSLOT, int DP0>
DI void quad(Ctx& c, const Frag4& a, const Frag4& b, Frag4& dst) { quad_<ROLE, VAR, Z, AH, BH, RSLOT, RISB, DSLOT, DP0>(c, a, b, dst, mkseq<16>{}); }
template <int ROLE> DI void epoch_end(Ctx& c, unsigned nxt, bool novm = false) {
  if constexpr (ROLE == R_LOAD) {
    c.sA += 16384; if (--c.left == 0) { c.sA = nxt; c.left = c.nleft; }
    if (!novm) asm volatile("s_waitcnt vmcnt(32)" ::: "memory");
  }
  asm volatile("s_waitcnt lgkmcnt(0)\n\ts_barrier" ::: "memory");
  GAP();
}
DI void lds_st(Ctx& c, unsigned off, unsigned v) { *(volatile LAS unsigned*)(size_t)(c.ldsb + off) = v; }
DI unsigned lds_ld(Ctx& c, unsigned off) { return *(volatile LAS unsigned*)(size_t)(c.ldsb + off); }

DI void poll_addr(Ctx& c, int ch, int j, int ti) {
  const unsigned line = c.lane < 32 ? 128u + (unsigned)j * 8u : (unsigned)ti * 8u, l = (unsigned)c.lane & 31u;
  c.pa = WS_FLG + (ch < 2 ? (line + (l >> 2)) * 128u + c.rank * 16u + (l & 3u) * 4u : (line + (unsigned)(4 * ch - 5) / 7u) * 128u + l * 4u);
}
DI void wait_flags(Ctx& c, const char* addr) {
  const unsigned long long t0 = __builtin_amdgcn_s_memrealtime();
  for (;;) {
    if (__all((int)(ld_sys(addr) - c.call) >= 0)) return;
    if (rfl(__hip_atomic_load((const unsigned*)(c.wsme + WS_TMO), __ATOMIC_RELAXED, __HIP_MEMORY_SCOPE_AGENT)) == c.call ||
        __builtin_amdgcn_s_memrealtime() - t0 > DEADLINE) {
      if (c.lane == 0) { __hip_atomic_store((unsigned*)(c.wsme + WS_TMO), c.call, __ATOMIC_RELAXED, __HIP_MEMORY_SCOPE_AGENT); lds_st(c, L_BAD, 1u); }
      return;
    }
    __builtin_amdgcn_s_sleep(2);
  }
}
template <int ROLE> DI void gate(Ctx& c) {
  if constexpr (ROLE == R_POLL) {
    if (__builtin_expect(!c.ok, 0)) wait_flags(c, c.wsme + c.pa);
  }
}

template <int ROLE, bool FIRST>
DI void iter(Ctx& c, Frag4& A0, Frag4& A1, Frag4& BX, Frag4& BY, bool novm, unsigned nxt) {
  quad<ROLE, 0, FIRST, 0, 0, 1, true, 0, 0>(c, A0, BX, BY);
  quad<ROLE, 0, FIRST, 0, 1, 1, false, 0, 8>(c, A0, BY, A1);
  epoch_end<ROLE>(c, nxt, novm);
  quad<ROLE, 1, FIRST, 1, 1, 2, false, 1, 0>(c, A1, BY, A0);
  quad<ROLE, 0, FIRST, 1, 0, 2, true, 1, 8>(c, A1, BX, BY);
  epoch_end<ROLE>(c, nxt, novm);
  quad<ROLE, 0, false, 0, 0, 3, true, 2, 0>(c, A0, BY, BX);
  quad<ROLE, 0, false, 0, 1, 3, false, 2, 8>(c, A0, BX, A1);
  epoch_end<ROLE>(c, nxt);
  quad<ROLE, 0, false, 1, 1, 0, false, 3, 0>(c, A1, BX, A0);
  quad<ROLE, 2, false, 1, 0, 0, true, 3, 8>(c, A1, BY, BX);
  gate<ROLE>(c);
  epoch_end<ROLE>(c, nxt);
}

struct Seg { int j, c0, c1; };
DI Seg segment(int q, int g, int nt, int np) {
  if (g < np) return Seg{jmap(q, nt - 1 - g), 0, 2};
  const int r = g - np;
  return Seg{jmap(q, r), r >= nt - np ? 2 : 0, 16};
}

template <int ROLE>
DI void body(Ctx& c, float* __restrict__ out, float scale, int ti, int q, int wr, int wc) {
  Frag4 A0, A1, BX, BY;
  const int nt = ntile(q), np = q < 4 ? NP5 : NP4, nseg = nt + np;
  auto src = [&](const Seg& g) { return c.stg + (wc == 0 ? (unsigned)ti << 20 : WB + ((unsigned)g.j << 20)) + (unsigned)g.c0 * 65536u; };
  Seg cur = segment(q, 0, nt, np);
  c.left = (unsigned)(cur.c1 - cur.c0) * 4u - 4u; c.sA = src(cur);
  if constexpr (ROLE == R_POLL) { c.ok = false; poll_addr(c, 0, cur.j, ti); }
  gate<ROLE>(c);
  asm volatile("s_waitcnt lgkmcnt(0)\n\ts_barrier" ::: "memory"); GAP();
  if constexpr (ROLE == R_LOAD) {
    dma_all<0>(c, mkseq<16>{}); c.sA += 16384;
    dma_all<1>(c, mkseq<16>{}); c.sA += 16384;
    dma_all<2>(c, mkseq<16>{}); c.sA += 16384;
    dma_all<3>(c, mkseq<16>{}); c.sA += 16384;
    asm volatile("s_waitcnt vmcnt(32)" ::: "memory");
  }
  asm volatile("s_barrier" ::: "memory"); GAP();
  rd_all<0, false>(c, A0, mkseq<8>{}); rd_all<0, true>(c, BX, mkseq<8>{});
  asm volatile("s_waitcnt lgkmcnt(0)\n\ts_barrier" ::: "memory"); GAP();
#pragma unroll 1
  for (int g = 0; g < nseg; ++g) {
    const Seg nx = segment(q, g + 1 < nseg ? g + 1 : g, nt, np);
    const unsigned nxt = src(nx);
    c.nleft = (unsigned)(nx.c1 - nx.c0) * 4u;
    auto aim = [&](int ch) { if constexpr (ROLE == R_POLL) { const int c2 = ch + 2; if (c2 < cur.c1) poll_addr(c, c2, cur.j, ti); else poll_addr(c, nx.c0 + c2 - cur.c1, nx.j, ti); } };
    aim(cur.c0);
    iter<ROLE, true>(c, A0, A1, BX, BY, g > 0, nxt);
#pragma unroll 1
    for (int ch = cur.c0 + 1; ch < cur.c1; ++ch) {
      aim(ch);
      iter<ROLE, false>(c, A0, A1, BX, BY, false, nxt);
    }
    asm volatile("s_nop 15" ::: "memory"); GAP();
    const float sc = lds_ld(c, L_BAD) ? __builtin_nanf("") : scale;
    float* o = out + (size_t)(ti * 256 + wr * 128 + 4 * (c.lane >> 4)) * NN + cur.j * 256 + wc * 128 + 4 * (c.lane & 15);
    if (cur.c0 != 0) out_add(o, sc);
    else if (cur.c1 != 16) out_all<true>(o, sc, mkseq<64>{});
    else out_all<false>(o, sc, mkseq<64>{});
    GAP();
    cur = nx;
  }
  asm volatile("s_waitcnt vmcnt(0)" ::: "memory");
}

DI void unit_load(const Params& p, unsigned item, unsigned t, int lane, unsigned (&d)[16]) {
  const unsigned ks = item & 7u, strip = item >> 3, sub = (t >> 1) * 128u + (t & 1u) * 64u, g = lane >> 4, c = lane & 15;
  if (strip < 16u) {
    const unsigned char* s = p.x + (size_t)(strip * 256u + sub + c) * KS + ks * 64u + g * 16u;
#pragma unroll
    for (int f = 0; f < 4; ++f) { const i32x4 v = *(const i32x4*)(s + f * 16 * KS); d[4 * f] = v[0]; d[4 * f + 1] = v[1]; d[4 * f + 2] = v[2]; d[4 * f + 3] = v[3]; }
  } else {
    const unsigned char* s = p.w + (size_t)(ks * 64u + g * 16u) * NN + (strip - 16u) * 256u + sub + c * 4u;
#pragma unroll
    for (int i = 0; i < 16; ++i) d[i] = *(const unsigned*)(s + (size_t)i * NN);
  }
}
DI void unit_fix(unsigned item, const unsigned (&d)[16], i32x4 (&o)[4]) {
  if (item < 128u) {
#pragma unroll
    for (int f = 0; f < 4; ++f) o[f] = (i32x4){(int)d[4 * f], (int)d[4 * f + 1], (int)d[4 * f + 2], (int)d[4 * f + 3]};
  } else {
#pragma unroll
    for (int a = 0; a < 4; ++a) {
      const unsigned t0 = __builtin_amdgcn_perm(d[4 * a + 1], d[4 * a], 0x05010400u), t1 = __builtin_amdgcn_perm(d[4 * a + 1], d[4 * a], 0x07030602u);
      const unsigned t2 = __builtin_amdgcn_perm(d[4 * a + 3], d[4 * a + 2], 0x05010400u), t3 = __builtin_amdgcn_perm(d[4 * a + 3], d[4 * a + 2], 0x07030602u);
      o[0][a] = (int)__builtin_amdgcn_perm(t2, t0, 0x05040100u); o[1][a] = (int)__builtin_amdgcn_perm(t2, t0, 0x07060302u);
      o[2][a] = (int)__builtin_amdgcn_perm(t3, t1, 0x05040100u); o[3][a] = (int)__builtin_amdgcn_perm(t3, t1, 0x07060302u);
    }
  }
}
DI rsrc_t mk_rsrc(char* base) { return __builtin_amdgcn_make_buffer_rsrc(base, 0, WS_USED, 0x00020000); }

DI void pusher(const Params& p, Ctx& c, int pi, int wid, unsigned rank, unsigned cnt) {
  const int lane = c.lane, pr = wid & 1;
  const unsigned sid = (unsigned)(pi * 2 + pr);
  const unsigned ring = c.ldsb + pr * (RING * 4096) + c.voff, full = L_FULL + pr * 4, fre = L_FREE + pr * 4, hdr = L_HDR + pr * 64;
  if (wid < 2) {
    unsigned d[16], tk = sid, item = (sid >> 2) * 8u;
    unit_load(p, item, tk & 3u, lane, d);
    const unsigned ord = L_ORD + pr * 2048;
#pragma unroll
    for (int i = 0; i < 5; ++i) lds_st(c, ord + (i * 64 + lane) * 4, ((const unsigned*)p.order)[i * 64 + lane]);
    const unsigned base = rfl(cnt) * NUNIT - NSEND;
#pragma unroll 1
    for (int n = 0;; ++n) {
      unsigned nx = 0;
      if (lane == 0) nx = __hip_atomic_fetch_add((unsigned*)(c.wsme + WS_TKT), 1u, __ATOMIC_RELAXED, __HIP_MEMORY_SCOPE_AGENT);
      if (n > 0) unit_load(p, item, tk & 3u, lane, d);
      i32x4 o[4]; unit_fix(item, d, o);
      while ((int)rfl(lds_ld(c, fre)) < n + 1 - RING) __builtin_amdgcn_s_sleep(2);
#pragma unroll
      for (int f = 0; f < 4; ++f) *(LAS i32x4*)(size_t)(ring + (n % RING) * 4096 + f * 1024) = o[f];
      if (lane == 0) lds_st(c, hdr + (n % RING) * 4, item * 4u + (tk & 3u));
      asm volatile("s_waitcnt lgkmcnt(0)" ::: "memory");
      if (lane == 0) lds_st(c, full, (unsigned)(n + 1));
      tk = rfl(nx) - base;
      if (tk >= NUNIT) { if (lane == 0) lds_st(c, full, (unsigned)(n + 1) | 0x10000u); break; }
      item = rfl(*(volatile LAS unsigned short*)(size_t)(c.ldsb + ord + (tk >> 2) * 2u));
      item = item < (unsigned)NITEM ? item : 0u;
    }
  } else {
    rsrc_t dst[7];
#pragma unroll
    for (int i = 0; i < 7; ++i) dst[i] = mk_rsrc(p.ws[(rank + 1u + (i + sid) % 7u) & 7u]);
    char* fbase = p.ws[(rank + 1u + (unsigned)(lane < 7 ? lane : 0)) & 7u] + WS_FLG + rank * 16u;
    c.call = rfl(cnt) + 1u; c.stg = WS_STG + (c.call & 1u) * STG_BYTES;
    unsigned prev = 0;
#pragma unroll 1
    for (unsigned n = 0;; ++n) {
      unsigned fu;
      while (((fu = rfl(lds_ld(c, full))) & 0xffffu) <= n && fu < 0x10000u) __builtin_amdgcn_s_sleep(1);
      if ((fu & 0xffffu) <= n) break;
      i32x4 o[4];
#pragma unroll
      for (int f = 0; f < 4; ++f) o[f] = *(const LAS i32x4*)(size_t)(ring + (n % RING) * 4096 + f * 1024);
      const unsigned u = rfl(lds_ld(c, hdr + (n % RING) * 4));
      unsigned so[7];
#pragma unroll
      for (int i = 0; i < 7; ++i) so[i] = c.stg + unit_off(u >> 5, 14u + 7u * ((u >> 2) & 7u) - (i + sid) % 7u, u & 3u);
      asm volatile("s_waitcnt lgkmcnt(0)\n\ts_nop 4" : "+v"(o[0]), "+v"(o[1]), "+v"(o[2]), "+v"(o[3]) :: "memory");
      if (lane == 0) lds_st(c, fre, n + 1u);
#pragma unroll
      for (int f = 0; f < 4; ++f) {
#pragma unroll
        for (int i = 0; i < 7; ++i) st_wt(o[f], c.voff, dst[i], so[i] + f * 1024);
        asm volatile("s_waitcnt vmcnt(%0)" :: "i"(PUSH_W) : "memory");
        if (f == (PUSH_W + 6) / 7 - 1 && n > 0) {
          if (lane < 7) st_sys(fbase + prev, c.call);
          if ((n & 1) == 0) STAMP((n >> 1) - 1);
        }
      }
      prev = (u >> 2) * 128u + (u & 3u) * 4u;
    }
    asm volatile("s_waitcnt vmcnt(0)" ::: "memory");
    if (lane < 7) st_sys(fbase + prev, c.call);
    STAMP(55);
  }
}

__global__ void __launch_bounds__(256, 1) fused_gemm_gather(Params p) {
  extern __shared__ __attribute__((aligned(1024))) char smem[];
  const int tid = threadIdx.x, wid = __builtin_amdgcn_readfirstlane(tid >> 6), lane = tid & 63, wr = wid >> 1, wc = wid & 1;
  const int b = blockIdx.x, xc = b & 7, s = b >> 3;
  const int u60 = (xc & 1) * 30 + s - 2, ti = (xc >> 1) * 4 + (u60 & 3), q = u60 >> 2;
  const unsigned rank = (unsigned)p.rank & 7u;
  Ctx c;
  c.lane = lane; c.wsme = p.ws[rank]; c.rank = rank;
  const unsigned cnt = ld_sys(c.wsme + WS_CNT + b * 128);
  c.ldsb = (unsigned)(size_t)(LAS char*)smem;
  c.voff = lane * 16;
  if (tid < 16) lds_st(c, L_RDY + tid * 4, 0u);
  asm volatile("s_waitcnt lgkmcnt(0)\n\ts_barrier" ::: "memory");
  if (s < 2) {
    pusher(p, c, b, wid, rank, cnt);
    c.call = rfl(cnt) + 1u;
  } else {
    c.aoff = c.ldsb + c.voff + wr * 4096; c.boff = c.ldsb + c.voff + wc * 4096; c.aoffH = c.aoff + 65536; c.boffH = c.boff + 65536;
    asm volatile("" : "+v"(c.aoff), "+v"(c.boff), "+v"(c.aoffH), "+v"(c.boffH));
    c.m0base = c.ldsb + wc * 16384;
    c.rs = mk_rsrc(c.wsme);
    const float scale = p.sx[0] * p.sw[0];
    {
      const unsigned g = (unsigned)(q * 16 + ti);
      unsigned d[3][16];
#pragma unroll
      for (int i = 0; i < 3; ++i) { const unsigned it = g + 240u * i; unit_load(p, it < (unsigned)NITEM ? it : g, (unsigned)wid, lane, d[i]); }
      c.call = rfl(cnt) + 1u; c.stg = WS_STG + (c.call & 1u) * STG_BYTES;
#pragma unroll
      for (int i = 0; i < 3; ++i) {
        const unsigned it = g + 240u * i;
        if (it < (unsigned)NITEM) {
          i32x4 o[4]; unit_fix(it, d[i], o);
          asm volatile("s_nop 4" ::: "memory");
#pragma unroll
          for (int f = 0; f < 4; ++f) st_wt(o[f], c.voff, c.rs, c.stg + unit_off(it >> 3, it & 7u, (unsigned)wid) + f * 1024);
        }
      }
      asm volatile("s_waitcnt vmcnt(0)" ::: "memory");
      const unsigned it = g + 240u * (unsigned)lane;
      if (lane < 3 && it < (unsigned)NITEM) st_sys(c.wsme + WS_FLG + it * 128u + rank * 16u + wid * 4, c.call);
    }
    STAMP(0);
    if (wid < 2) body<R_LOAD>(c, p.out, scale, ti, q, wr, wc);
    else if (wid == 2) body<R_POLL>(c, p.out, scale, ti, q, wr, wc);
    else body<R_NONE>(c, p.out, scale, ti, q, wr, wc);
  }
  __syncthreads();
  if (tid == 0) st_sys(c.wsme + WS_CNT + b * 128, c.call);
}

static void make_order(unsigned short* order) {
  struct It { double key; int item; };
  It v[NITEM]; int n = 0;
  for (int ks = 0; ks < 8; ++ks) {
    for (int i = 0; i < 16; ++i) v[n++] = {ks - 100.0 + i * 1e-3, i * 8 + ks};
    for (int q = 0; q < 4; ++q) v[n++] = {ks - 100.0 + 0.5 + q * 1e-3, 128 + jmap(q, 0) * 8 + ks};
  }
  for (int q = 0; q < 15; ++q)
    for (int r = (q < 4 ? 1 : 0); r < ntile(q); ++r)
      for (int ks = 0; ks < 8; ++ks) {
        const int m = (r - (q < 4 ? 1 : 0)) * 8 + ks;
        const double ph = q < 4 ? PH5 : (double)((q - 4) * 4 % 11) / 11.0;
        v[n++] = {m + SKEW * ph + q * 1e-3, 128 + jmap(q, r) * 8 + ks};
      }
  std::stable_sort(v, v + n, [](const It& a, const It& b) { return a.key < b.key; });
  for (int i = 0; i < NITEM; ++i) order[i] = (unsigned short)v[i].item;
}

extern "C" void kernel_launch(int rank, int nranks, ncclComm_t comm, hipStream_t stream,
                              void* const* d_in, const int* in_sizes, int n_in,
                              void* d_out, int out_size, const int* in_dtypes, int out_dtype,
                              void* const* d_out_all, void* const* d_ws_all, size_t ws_bytes) {
  (void)comm; (void)in_dtypes; (void)out_dtype; (void)d_out_all;
  if (nranks != NR || n_in != 4 || (unsigned)rank >= (unsigned)NR || ws_bytes < (size_t)WS_USED || in_sizes[0] != MM * KS || in_sizes[1] != KS * NN || out_size != MM * NN) {
    fprintf(stderr, "kernel_launch: unexpected problem shape\n"); return;
  }
  HIP_CHECK(hipFuncSetAttribute((const void*)fused_gemm_gather, hipFuncAttributeMaxDynamicSharedMemorySize, (int)LDS_BYTES));
  Params p;
  p.x = (const unsigned char*)d_in[0]; p.w = (const unsigned char*)d_in[1]; p.sx = (const float*)d_in[2]; p.sw = (const float*)d_in[3];
  p.out = (float*)d_out; p.rank = rank; p.pad = 0;
  for (int i = 0; i < NR; ++i) p.ws[i] = (char*)d_ws_all[i];
  struct Order { unsigned short v[NITEM]; Order() { make_order(v); } };
  static const Order ord;
  std::copy(ord.v, ord.v + NITEM, p.order);
  fused_gemm_gather<<<dim3(256), dim3(256), LDS_BYTES, stream>>>(p);
  HIP_CHECK(hipGetLastError());
}
```

```cpp
#include <hip/hip_runtime.h>
#include <rccl/rccl.h>
#include <cstdio>
#include <utility>
#include <algorithm>

#define HIP_CHECK(expr) do { hipError_t _e = (expr); if (_e != hipSuccess) { \
    fprintf(stderr, "HIP error %s at %s:%d\n", hipGetErrorString(_e), __FILE__, __LINE__); \
    return; } } while (0)

#define DI __device__ __forceinline__
#define HD __host__ __device__ __forceinline__
#define LAS __attribute__((address_space(3)))
#define GAP() __builtin_amdgcn_sched_barrier(0)
typedef int i32x4 __attribute__((ext_vector_type(4)));
typedef int i32x8 __attribute__((ext_vector_type(8)));
typedef float f32x4 __attribute__((ext_vector_type(4)));
typedef __amdgpu_buffer_rsrc_t rsrc_t;
#define AC10(p) "a" #p "0", "a" #p "1", "a" #p "2", "a" #p "3", "a" #p "4", "a" #p "5", "a" #p "6", "a" #p "7", "a" #p "8", "a" #p "9"
#define ACLOB "a0", "a1", "a2", "a3", "a4", "a5", "a6", "a7", "a8", "a9", AC10(1), AC10(2), AC10(3), AC10(4), AC10(5), AC10(6), AC10(7), AC10(8), AC10(9), \
  AC10(10), AC10(11), AC10(12), AC10(13), AC10(14), AC10(15), AC10(16), AC10(17), AC10(18), AC10(19), AC10(20), AC10(21), AC10(22), AC10(23), AC10(24), \
  "a250", "a251", "a252", "a253", "a254", "a255"
template <int... I> using iseq = std::integer_sequence<int, I...>;
template <int N> using mkseq = std::make_integer_sequence<int, N>;

constexpr int NR = 8, MM = 4096, NN = 16384, KS = 512;
constexpr unsigned WS_CNT = 0;
constexpr unsigned WS_TMO = 32u << 10;
constexpr unsigned WS_TKT = WS_TMO + 128u;
constexpr unsigned WS_ROT = WS_TMO + 256u;
constexpr unsigned WS_SHR = WS_TMO + 512u;
constexpr unsigned WS_DON = WS_TMO + 1024u;
constexpr unsigned WS_FLG = 64u << 10;
constexpr unsigned WS_STG = 1u << 20;
constexpr unsigned STG_BYTES = 80u << 20, WB = 16u << 20;
constexpr unsigned WS_USED = WS_STG + 2u * STG_BYTES;
constexpr int NITEM = 640;
constexpr unsigned L_POL = 128u << 10, L_RDY = L_POL + 1024u, L_BAD = L_RDY + 4u, L_FULL = L_RDY + 16u, L_FREE = L_RDY + 32u, LDS_BYTES = L_RDY + 64u;
constexpr unsigned DEADLINE = 50000000u;
constexpr int RING = 2;
constexpr unsigned NSEND = 32, NUNIT = NITEM * 4, NALL = NUNIT + 256, L_ORD = 64u << 10, L_HDR = 68u << 10;
constexpr int PUSH_W = 12;
constexpr int LAG = 3;
constexpr unsigned OOB = 0xC0000000u;

struct Params { const unsigned char* x; const unsigned char* w; const float* sx; const float* sw; float* out[NR]; char* ws[NR]; int rank; int pad; unsigned short order[NITEM]; };

HD int jmap(int q, int r) { return 4 * q + ((r + q) & 3); }
DI unsigned now() { return (unsigned)__builtin_amdgcn_s_memrealtime(); }
DI unsigned rfl(unsigned v) { return __builtin_amdgcn_readfirstlane(v); }
DI unsigned ld_sys(const void* p) { return __hip_atomic_load((const unsigned*)p, __ATOMIC_RELAXED, __HIP_MEMORY_SCOPE_SYSTEM); }
DI void st_sys(void* p, unsigned v) { __hip_atomic_store((unsigned*)p, v, __ATOMIC_RELAXED, __HIP_MEMORY_SCOPE_SYSTEM); }
DI unsigned unit_off(unsigned strip, unsigned pos, unsigned t) { return (strip << 20) + (pos >> 1) * 32768u + (t & 1u) * 16384u + (pos & 1u) * 8192u + (t >> 1) * 4096u; }

struct Frag4 { i32x4 lo[4], hi[4]; };

struct Ctx {
  unsigned aoff, boff, aoffH, boffH, voff, ovoff, ldsb;
  rsrc_t rs, ro;
  unsigned sA, sB[2], sd, m0base, left, nA, nB;
  unsigned call, stg, so[2], msk; int lane, wid; float sc;
  char* wsme;
  unsigned pv, pvn, ps, gch, pl, fv, pa; unsigned long long chk, w0;
  f32x4 pk[64];
};
DI void lds_st(Ctx& c, unsigned off, unsigned v) { *(volatile LAS unsigned*)(size_t)(c.ldsb + off) = v; }
DI unsigned lds_ld(Ctx& c, unsigned off) { return *(volatile LAS unsigned*)(size_t)(c.ldsb + off); }

template <int SLOT, bool ISB, int I> DI void rd1(Ctx& c, Frag4& f) {
  constexpr int off = (SLOT & 1) * 32768 + (ISB ? 16384 : 0) + (I & 1) * 8192 + (I >> 1) * 1024;
  const unsigned base = SLOT >= 2 ? (ISB ? c.boffH : c.aoffH) : (ISB ? c.boff : c.aoff);
  const i32x4 v = *(const LAS i32x4*)((LAS char*)(size_t)base + off);
  if constexpr (I & 1) f.hi[I >> 1] = v; else f.lo[I >> 1] = v;
}
DI i32x8 cat(i32x4 a, i32x4 b) { return __builtin_shufflevector(a, b, 0, 1, 2, 3, 4, 5, 6, 7); }
template <bool ZERO, int AH, int BH, int G> DI void mma1(const Frag4& a, const Frag4& b) {
  constexpr int i = G >> 2, j = G & 3, r = 4 * (8 * (AH * 4 + i) + BH * 4 + j);
  if constexpr (ZERO)
    asm volatile("v_mfma_f32_16x16x128_f8f6f4 a[%c2:%c3], %0, %1, 0 blgp:1" :: "v"(cat(a.lo[i], a.hi[i])), "v"(cat(b.lo[j], b.hi[j])), "i"(r), "i"(r + 3) : ACLOB);
  else
    asm volatile("v_mfma_f32_16x16x128_f8f6f4 a[%c2:%c3], %0, %1, a[%c2:%c3] blgp:1" :: "v"(cat(a.lo[i], a.hi[i])), "v"(cat(b.lo[j], b.hi[j])), "i"(r), "i"(r + 3) : ACLOB);
}
template <int SLOT, int ISB, int P> DI void dma1(Ctx& c) {
  if constexpr (P == 0)
    asm volatile("s_add_u32 m0, %0, %1\n\ts_nop 0\n\tbuffer_load_dwordx4 %2, %3, %4 offen sc0 sc1 lds" :: "s"(c.m0base), "i"(SLOT * 32768 + ISB * 16384), "v"(c.voff), "s"(c.rs), "s"(c.sd) : "memory", "scc");
  else
    asm volatile("buffer_load_dwordx4 %0, %1, %2 offen offset:%3 sc0 sc1 lds" :: "v"(c.voff), "s"(c.rs), "s"(c.sd), "i"(P * 1024) : "memory");
}
template <int SLOT, int... P> DI void dma_all(Ctx& c, iseq<P...>) { c.sd = c.sA + SLOT * 16384; (dma1<SLOT, 0, P>(c), ...); c.sd = c.sB[SLOT & 1] + SLOT * 16384; (dma1<SLOT, 1, P>(c), ...); }
template <int SLOT, bool ISB, int... I> DI void rd_all(Ctx& c, Frag4& f, iseq<I...>) { (rd1<SLOT, ISB, I>(c, f), ...); }
DI void st_wt(const i32x4& d, unsigned voff, const rsrc_t& r, unsigned soff) {
  asm volatile("buffer_store_dwordx4 %0, %1, %2, %3 offen sc0 sc1\n\ts_nop 1" :: "v"(d), "v"(voff), "s"(r), "s"(soff) : "memory");
}

constexpr int oAH(int n) { return (n >> 4) & 1; }
constexpr int oBH(int n) { return n >> 5; }
constexpr int oreg(int n) { return 4 * (8 * (oAH(n) * 4 + ((n >> 2) & 3)) + oBH(n) * 4) + (n & 3); }
constexpr int XH = 8 * LAG + 1, NX = XH + 26;
constexpr int cr(int x) { return x < 0 ? 0 : x < 6 ? (16 * x + 17) / 3 : x < XH ? 32 : x == XH ? 36 : x == XH + 1 ? 42 : x == XH + 2 ? 54 : 64; }
constexpr int cs(int x) {
  return x < 0 ? 0 : x < 6 ? 2 * x + 2 : x < XH ? 12 + (20 * (x - 5) + XH - 7) / (XH - 6) : x < XH + 4 ? 32 + 3 * (x - XH + 1) : x < NX ? 44 + (20 * (x - XH - 3) + NX - XH - 5) / (NX - XH - 4) : 64;
}
struct Plan { short a[NX * 16 + 1], m[NX * 16 + 1], s[NX * 16 + 1]; };
constexpr int base_cost(int g) { return 6 + (g < 8 ? 5 : (g & 1) == 0 ? 8 : 0) + (g == 6 || g == 8 ? 6 : 0); }
constexpr Plan make_plan() {
  Plan p{}; int a = 0, m = 0, s = 0;
  for (int x = 0; x < NX; ++x) {
    const int ae = 4 * cr(x), se = cs(x), ns = se - cs(x - 1), span = 16;
    int me = 4 * cs(x + 1); if (me > ae) me = ae;
    for (int g = 0; g < 16; ++g) {
      const int want = cs(x - 1) + (g + 1 >= span ? ns : ((g + 1) * ns) / span);
      int load = base_cost(g);
      int rem = 4 * (ae - a) + 4 * (me - m) + 34 * (se - s); for (int h = g; h < 16; ++h) rem += base_cost(h);
      int target = rem / (16 - g); if (target < 30) target = 30;
      if (s < want && m >= 4 * s + 4) { ++s; load += 34; }
      for (;;) {
        const bool urgent = s < want && m < 4 * s + 4;
        if (g < 15 && load + 4 > target && !(urgent && load < 40)) break;
        if (m < me && m < a && (urgent || (me - m) >= (ae - a) || a >= ae)) { ++m; load += 4; continue; }
        if (a < ae) { ++a; load += 4; continue; }
        if (m < me && m < a) { ++m; load += 4; continue; }
        break;
      }
      if (g == 15) s = se;
      p.a[x * 16 + g + 1] = (short)a; p.m[x * 16 + g + 1] = (short)m; p.s[x * 16 + g + 1] = (short)s;
    }
  }
  return p;
}
constexpr Plan PLAN = make_plan();
template <int I> DI void op_a(Ctx& c) { asm volatile("v_accvgpr_read_b32 %0, a%c1" : "=v"(c.pk[I >> 2][I & 3]) : "i"(oreg(I >> 2) + 4 * (I & 3))); }
template <int I> DI void op_m(Ctx& c) { asm volatile("v_mul_f32 %0, %1, %0" : "+v"(c.pk[I >> 2][I & 3]) : "s"(c.sc)); }
template <int n, bool WT = false> DI void op_s(Ctx& c) {
  const unsigned so = c.so[oBH(n)] + (unsigned)(16 * (oAH(n) * 4 + ((n >> 2) & 3)) + (n & 3)) * (NN * 4u);
  if constexpr (WT) asm volatile("buffer_store_dwordx4 %0, %1, %2, %3 offen sc1\n\ts_nop 1" :: "v"(c.pk[n]), "v"(c.ovoff), "s"(c.ro), "s"(so) : "memory");
  else asm volatile("buffer_store_dwordx4 %0, %1, %2, %3 offen nt\n\ts_nop 1" :: "v"(c.pk[n]), "v"(c.ovoff), "s"(c.ro), "s"(so) : "memory");
}
template <int I0, int... I> DI void ops_a(Ctx& c, iseq<I...>) { (op_a<I0 + I>(c), ...); }
template <int I0, int... I> DI void ops_m(Ctx& c, iseq<I...>) { (op_m<I0 + I>(c), ...); }
template <int I0, int... I> DI void ops_s(Ctx& c, iseq<I...>) { (op_s<I0 + I>(c), ...); }
template <int X, int G> DI void epi_gap(Ctx& c) {
  if constexpr (X >= 0 && X < NX) {
    constexpr int i = X * 16 + G;
    static_assert(G < 15 || (PLAN.a[i + 1] == 4 * cr(X) && PLAN.s[i + 1] == cs(X)), "plan incomplete");
    static_assert(PLAN.m[i + 1] <= PLAN.a[i + 1] && 4 * PLAN.s[i + 1] <= PLAN.m[i + 1], "plan order");
    ops_a<PLAN.a[i]>(c, mkseq<PLAN.a[i + 1] - PLAN.a[i]>{});
    ops_m<PLAN.m[i]>(c, mkseq<PLAN.m[i + 1] - PLAN.m[i]>{});
    ops_s<PLAN.s[i]>(c, mkseq<PLAN.s[i + 1] - PLAN.s[i]>{});
  }
}
template <int XD, int n> DI void flush1(Ctx& c) {
  constexpr int a0 = PLAN.a[XD * 16] > 4 * n ? PLAN.a[XD * 16] : 4 * n, m0 = PLAN.m[XD * 16] > 4 * n ? PLAN.m[XD * 16] : 4 * n;
  if constexpr (a0 < 4 * n + 4) ops_a<a0>(c, mkseq<4 * n + 4 - a0>{});
  if constexpr (m0 < 4 * n + 4) ops_m<m0>(c, mkseq<4 * n + 4 - m0>{});
  op_s<n, true>(c);
}
template <int XD, int... I> DI void flush(Ctx& c, iseq<I...>) { (flush1<XD, cs(XD - 1) + I>(c), ...); }

template <int n> DI void shr1(Ctx& c, unsigned so0) {
  f32x4 v;
  asm volatile("v_accvgpr_read_b32 %0, a%c5\n\tv_accvgpr_read_b32 %1, a%c6\n\tv_accvgpr_read_b32 %2, a%c7\n\tv_accvgpr_read_b32 %3, a%c8\n\t"
               "v_mul_f32 %0, %4, %0\n\tv_mul_f32 %1, %4, %1\n\tv_mul_f32 %2, %4, %2\n\tv_mul_f32 %3, %4, %3"
               : "=v"(v[0]), "=v"(v[1]), "=v"(v[2]), "=v"(v[3]) : "s"(c.sc), "i"(oreg(n)), "i"(oreg(n) + 4), "i"(oreg(n) + 8), "i"(oreg(n) + 12));
  asm volatile("buffer_store_dwordx4 %0, %1, %2, %3 offen sc0 sc1\n\ts_nop 1" :: "v"(v), "v"(c.ovoff), "s"(c.ro),
               "s"(so0 + (unsigned)(16 * (oAH(n) * 4 + ((n >> 2) & 3)) + (n & 3)) * (NN * 4u)) : "memory");
}
template <int... n> DI void shr_all(Ctx& c, unsigned so0, iseq<n...>) { (shr1<n>(c, so0), ...); }
template <int R> DI void acc_zero1() { asm volatile("v_accvgpr_write_b32 a%c0, 0" :: "i"(R) : ACLOB); }
template <int... R> DI void acc_zero(iseq<R...>) { (acc_zero1<R>(), ...); }

template <int T, bool Z, int X, int Q, int AH, int BH, bool RISB, int G>
DI void gap1(Ctx& c, const Frag4& a, const Frag4& b, Frag4& dst) {
  constexpr int DSLOT = Q >> 1, DISB = Q & 1, RSLOT = (DSLOT + 1) & 3;
  if constexpr (T != 0) {
    if constexpr (BH == 1) mma1<false, AH, BH, G>(a, b);
    if constexpr (G < 8 && (Q & 3) != 3 && !(T == 2 && Q >= 6)) rd1<RSLOT, RISB, G>(c, dst);
    if constexpr (T == 1 && G == 0) { c.sd = (DISB ? c.sB[DSLOT & 1] : c.sA) + DSLOT * 16384; asm volatile("" : "+s"(c.sd)); }
    if constexpr (T == 1 && G >= 1 && G < 5 && (Q & 3) != 1) dma1<DSLOT, DISB, G - 1>(c);
    epi_gap<X, G>(c);
    GAP();
    return;
  }
  mma1<Z, AH, BH, G>(a, b);
  if constexpr (G < 8) rd1<RSLOT, RISB, G>(c, dst);
  if constexpr (G == 6) { c.sd = (DISB ? c.sB[DSLOT & 1] : c.sA) + DSLOT * 16384; asm volatile("" : "+s"(c.sd)); }
  if constexpr (G >= 8 && (G & 1) == 0) dma1<DSLOT, DISB, ((G - 8) >> 1)>(c);
  if constexpr (Q == 0 && G == 3) {
    asm volatile("s_mov_b32 m0, %0\n\ts_nop 0\n\tbuffer_load_dword %1, %2, %3 offen sc0 sc1 lds" :: "s"(c.ldsb + L_POL + c.wid * 256), "v"(c.pv), "s"(c.rs), "s"(c.ps) : "memory");
  }
  if constexpr (Q == 6 && G == 11) asm volatile("s_add_u32 %0, %0, 0x10000\n\ts_cmp_eq_u32 %1, 1\n\ts_cselect_b32 %0, %2, %0" : "+s"(c.sA) : "s"(c.left), "s"(c.nA) : "scc");
  if constexpr (Q == 5 && G == 9) asm volatile("s_add_u32 %0, %0, 0x10000\n\ts_cmp_eq_u32 %1, 1\n\ts_cselect_b32 %0, %2, %0" : "+s"(c.sB[0]) : "s"(c.left), "s"(c.nB) : "scc");
  if constexpr (Q == 7 && G == 9) {
    unsigned t;
    asm volatile("s_cmp_eq_u32 %2, 1\n\ts_cselect_b32 %1, 0xfff10000, %4\n\ts_add_u32 %0, %0, %1\n\ts_cmp_eq_u32 %2, %5\n\ts_cselect_b32 %0, %3, %0"
                 : "+s"(c.sB[1]), "=&s"(t) : "s"(c.left), "s"(c.sB[0] | c.msk), "s"(0x10000u), "i"(17 - LAG) : "scc");
  }
  if constexpr (Q == 7 && G == 13) asm volatile("s_sub_u32 %0, %0, 1\n\ts_cmp_eq_u32 %0, 0\n\ts_cselect_b32 %0, 16, %0" : "+s"(c.left) :: "scc");
  if constexpr (Q == 6 && G == 9) c.fv = *(const LAS unsigned*)(size_t)c.pl;
  if constexpr (Q == 1 && G == 9) {
    asm volatile("v_add_u32 %0, %1, %2" : "=v"(c.pa) : "s"(c.ps), "v"(c.pv));
    asm volatile("s_cmp_gt_u32 %1, 1\n\ts_cselect_b64 %0, %2, 0" : "=s"(c.chk) : "s"(c.gch), "s"(c.w0) : "scc");
  }
  if constexpr (Q == 2 && G == 9) {
    unsigned long long m;
    asm volatile("s_add_u32 %0, %0, 1\n\ts_and_b32 %0, %0, 15\n\ts_cselect_b64 %1, 0, -1" : "+s"(c.gch), "=s"(m) :: "scc");
    asm volatile("v_cndmask_b32 %0, %0, %1, %2" : "+v"(c.pv) : "v"(c.pvn), "s"(m));
  }
  if constexpr (Q == 3 && G == 9)
    asm volatile("s_max_u32 %0, %1, 2\n\ts_mul_i32 %0, %0, 148\n\ts_add_u32 %0, %0, %2" : "=&s"(c.ps) : "s"(c.gch), "i"(2 * WS_FLG - 185) : "scc");
  if constexpr (Q == 3 && G == 11) asm volatile("s_lshr_b32 %0, %0, 8\n\ts_lshl_b32 %0, %0, 7" : "+s"(c.ps) :: "scc");
  if constexpr (Q == 7 && G == 11) { c.chk &= __builtin_amdgcn_ballot_w64((int)(c.fv - c.call) < 0); asm volatile("" : "+s"(c.chk)); }
  epi_gap<X, G>(c);
  GAP();
}
template <int T, bool Z, int X, int Q, int AH, int BH, bool RISB, int... G>
DI void quad_(Ctx& c, const Frag4& a, const Frag4& b, Frag4& dst, iseq<G...>) { (gap1<T, Z, X, Q, AH, BH, RISB, G>(c, a, b, dst), ...); }
template <int T, bool Z, int X, int Q, int AH, int BH, bool RISB>
DI void quad(Ctx& c, const Frag4& a, const Frag4& b, Frag4& dst) { quad_<T, Z, X, Q, AH, BH, RISB>(c, a, b, dst, mkseq<16>{}); }
template <int NV> DI void epoch_end() {
  constexpr int n = NV > 63 ? 63 : NV;
  __builtin_amdgcn_s_waitcnt((n & 15) | 0x70 | ((n >> 4) << 14));
  asm volatile("s_barrier" ::: "memory");
  GAP();
}
DI unsigned vmov(unsigned x) { unsigned v; asm volatile("v_mov_b32 %0, %1" : "=v"(v) : "s"(x)); return v; }
DI void wait_flags(Ctx& c, unsigned off) {
  const unsigned t0 = now();
  for (unsigned i = 1;; ++i) {
    if (__all((int)(__builtin_amdgcn_raw_buffer_load_b32(c.rs, off, 0, 17) - c.call) >= 0)) return;
    if ((i & 15u) == 0u &&
        (rfl(__builtin_amdgcn_raw_buffer_load_b32(c.rs, vmov(WS_TMO), 0, 17)) == c.call || now() - t0 > DEADLINE)) {
      __builtin_amdgcn_raw_buffer_store_b32(vmov(c.call), c.rs, vmov(WS_TMO), 0, 17);
      *(volatile LAS unsigned*)(size_t)vmov(c.ldsb + L_BAD) = vmov(0x7fc00000u);
      return;
    }
    asm volatile("s_sleep 6" ::: "memory");
  }
}
template <int ZB, int X0, int T = 0> DI void iter(Ctx& c, Frag4& A0, Frag4& A1, Frag4& BX, Frag4& BY) {
  constexpr int nd = T == 0 ? 16 : 12;
  constexpr auto nv = [](int x) { return cs(x) - cs(x - 4); };
  epoch_end<nd + nv(X0 - 1)>();
  quad<T, ZB == 0, X0 + 0, 0, 0, 0, true>(c, A0, BX, BY);
  quad<T, ZB == 1, X0 + 1, 1, 0, 1, false>(c, A0, BY, A1);
  epoch_end<(T == 2 ? 8 : nd) + nv(X0 + 1)>();
  quad<T, ZB == 1, X0 + 2, 2, 1, 1, false>(c, A1, BY, A0);
  quad<T, ZB == 0, X0 + 3, 3, 1, 0, true>(c, A1, BX, BY);
  epoch_end<(T == 2 ? 0 : nd) + nv(X0 + 3)>();
  quad<T, false, X0 + 4, 4, 0, 0, true>(c, A0, BY, BX);
  quad<T, false, X0 + 5, 5, 0, 1, false>(c, A0, BX, A1);
  if constexpr (T != 2) epoch_end<nd + nv(X0 + 5)>();
  quad<T, false, X0 + 6, 6, 1, 1, false>(c, A1, BX, A0);
  quad<T, false, X0 + 7, 7, 1, 0, true>(c, A1, BY, BX);
  if constexpr (T == 0) { if (__builtin_expect(c.chk != 0ull, 0)) wait_flags(c, c.pa); }
}

DI void body(Ctx& c, unsigned scale, int ti, int q, unsigned rank) {
  Frag4 A0, A1, BX, BY;
  const bool shr = q == 0 && (unsigned)(ti >> 3) == (rank & 1u);
  const int r0 = shr || q == 1 || q == 2 ? 0 : 1;
  const unsigned l31 = (unsigned)c.lane & 31u, hs = (rank >> 1) & 1u;
  auto strip = [&](int r) { return r == 0 ? (shr ? 62u + (rank >> 2) : 59u + q) : (unsigned)jmap(q, r); };
  auto lines = [&](int r) { return (c.lane < 32 || r > r0 ? 128u + strip(r) * 8u : (unsigned)ti * 8u) * 128u; };
  auto pvof = [&](int r) { return c.wid != 0 ? OOB : lines(r) + (r == 0 && shr && c.lane < 32 ? (l31 & 30u) | hs : l31) * 4u; };
  if (c.wid == 0) {
    for (int r = r0; r < 5; ++r) wait_flags(c, WS_FLG + lines(r) + (l31 >> 2) * 128u + rank * 16u + (l31 & 3u) * 4u);
  }
  asm volatile("s_waitcnt lgkmcnt(0)\n\ts_barrier" ::: "memory"); GAP();
  if (c.wid == 3) st_sys(c.wsme + WS_CNT + blockIdx.x * 128, c.call);
#pragma unroll 1
  for (unsigned dry = r0 == 1 ? OOB : 0u;; dry = 0u) {
    acc_zero(mkseq<256>{});
    c.nA = c.sA = (c.stg + ((unsigned)ti << 20) + c.wid * 4096) | dry; c.left = 15;
    c.sB[0] = (c.stg + WB + (strip(r0) << 20) + c.wid * 4096 + (shr ? hs * 16384u : 0u)) | dry; c.sB[1] = OOB;
    c.msk = shr ? OOB : 0u;
    dma_all<0>(c, mkseq<4>{}); dma_all<1>(c, mkseq<4>{}); dma_all<2>(c, mkseq<4>{}); dma_all<3>(c, mkseq<4>{});
    c.sA += 65536; c.sB[0] += 65536;
    asm volatile("s_cmp_eq_u32 %1, 0\n\ts_cselect_b64 %0, -1, 0" : "=s"(c.w0) : "s"(c.wid | dry) : "scc");
    c.pv = pvof(r0); c.gch = 2; c.ps = WS_FLG; c.pl = c.ldsb + L_POL + c.wid * 256 + c.lane * 4;
    asm volatile("s_waitcnt vmcnt(16)\n\ts_barrier" ::: "memory"); GAP();
    rd_all<0, false>(c, A0, mkseq<8>{}); rd_all<0, true>(c, BX, mkseq<8>{});
    auto next = [&](int r) { const int rn = r < 4 ? r + 1 : r; c.nB = (c.stg + WB + ((unsigned)jmap(q, rn) << 20) + c.wid * 4096) | dry; c.pvn = pvof(rn); };
    auto rest = [&](int r) __attribute__((always_inline)) {
      c.so[0] = r == 0 && shr ? OOB : ((unsigned)(ti * 256) * (NN * 4u) + strip(r) * 1024u) | dry; c.so[1] = c.so[0] + 512u;
      if (dry) c.left = 1;
#pragma unroll 1
      do {
        iter<-1, -100>(c, A0, A1, BX, BY);
      } while (c.left != 16u);
      c.sc = __uint_as_float(rfl(*(volatile LAS unsigned*)(size_t)vmov(c.ldsb + L_BAD)) | scale);
      iter<-1, -5>(c, A0, A1, BX, BY);
    };
    next(r0);
    rest(r0);
    if (shr) {
      unsigned so0 = (unsigned)(ti * 256) * (NN * 4u) + strip(0) * 1024u + hs * 512u;
      asm volatile("s_nop 15" : "+s"(so0) :: "memory");
      shr_all(c, so0, mkseq<32>{});
      c.msk = 0;
    }
#pragma unroll 1
    for (int r = dry ? 4 : r0 + 1; r < 5; ++r) {
      iter<0, 3>(c, A0, A1, BX, BY);
      iter<-1, 11>(c, A0, A1, BX, BY);
      iter<-1, 19>(c, A0, A1, BX, BY);
      next(r);
      if (r == 1 && shr) __builtin_amdgcn_raw_buffer_store_b32(vmov(c.call), c.rs, vmov(WS_SHR + ((ti & 7) * 4 + c.wid) * 4), 0, 17);
      iter<1, XH + 2>(c, A0, A1, BX, BY);
      iter<-1, XH + 10>(c, A0, A1, BX, BY);
      iter<-1, XH + 18>(c, A0, A1, BX, BY);
      rest(r);
    }
    iter<-1, 3, 1>(c, A0, A1, BX, BY);
    c.sA += 65536; c.sB[1] += 65536;
    iter<-1, 11, 1>(c, A0, A1, BX, BY);
    iter<-1, XH - 6, 2>(c, A0, A1, BX, BY);
    asm volatile("s_nop 15" ::: "memory"); GAP();
    flush<XH + 2>(c, mkseq<64 - cs(XH + 1)>{});
    asm volatile("fgg_code_end:" ::: "memory");
    if (!dry) break;
    asm volatile("s_waitcnt vmcnt(0) lgkmcnt(0)\n\ts_barrier" ::: "memory"); GAP();
  }
}

DI void unit_load(const Params& p, unsigned item, unsigned t, int lane, unsigned (&d)[16]) {
  const unsigned ks = item & 7u, strip = item >> 3, sub = (t >> 1) * 128u + (t & 1u) * 64u, g = lane >> 4, c = lane & 15;
  if (strip < 16u) {
    const unsigned char* s = p.x + (size_t)(strip * 256u + sub + c) * KS + ks * 64u + g * 16u;
#pragma unroll
    for (int f = 0; f < 4; ++f) { const i32x4 v = *(const i32x4*)(s + f * 16 * KS); d[4 * f] = v[0]; d[4 * f + 1] = v[1]; d[4 * f + 2] = v[2]; d[4 * f + 3] = v[3]; }
  } else {
    const unsigned char* s = p.w + (size_t)(ks * 64u + g * 16u) * NN + (strip - 16u) * 256u + (t & 1u) * 128u + (t >> 1) * 64u + c * 4u;
#pragma unroll
    for (int i = 0; i < 16; ++i) d[i] = *(const unsigned*)(s + (size_t)i * NN);
  }
}
DI void unit_fix(unsigned item, const unsigned (&d)[16], i32x4 (&o)[4]) {
  if (item < 128u || item >= (unsigned)NITEM) {
#pragma unroll
    for (int f = 0; f < 4; ++f) o[f] = (i32x4){(int)d[4 * f], (int)d[4 * f + 1], (int)d[4 * f + 2], (int)d[4 * f + 3]};
  } else {
#pragma unroll
    for (int a = 0; a < 4; ++a) {
      const unsigned t0 = __builtin_amdgcn_perm(d[4 * a + 1], d[4 * a], 0x05010400u), t1 = __builtin_amdgcn_perm(d[4 * a + 1], d[4 * a], 0x07030602u);
      const unsigned t2 = __builtin_amdgcn_perm(d[4 * a + 3], d[4 * a + 2], 0x05010400u), t3 = __builtin_amdgcn_perm(d[4 * a + 3], d[4 * a + 2], 0x07030602u);
      o[0][a] = (int)__builtin_amdgcn_perm(t2, t0, 0x05040100u); o[1][a] = (int)__builtin_amdgcn_perm(t2, t0, 0x07060302u);
      o[2][a] = (int)__builtin_amdgcn_perm(t3, t1, 0x05040100u); o[3][a] = (int)__builtin_amdgcn_perm(t3, t1, 0x07060302u);
    }
  }
}
DI rsrc_t mk_rsrc(char* base) { return __builtin_amdgcn_make_buffer_rsrc(base, 0, WS_USED, 0x00020000); }

DI unsigned shr_off(unsigned f, unsigned rank) { return ((rank & 1u) * 256u + f) * (8u * NN * 4u) + (62u * 256u + (rank >> 1) * 128u) * 4u; }
DI rsrc_t mk_out(float* base) { return __builtin_amdgcn_make_buffer_rsrc(base, 0, (unsigned)MM * NN * 4u, 0x00020000); }

DI void pusher(const Params& p, Ctx& c, int pi, int wid, unsigned rank, unsigned cnt) {
  const int lane = c.lane, pr = wid & 1;
  const unsigned sid = (unsigned)(pi * 2 + pr);
  const unsigned ring = c.ldsb + pr * (RING * 4096) + c.voff, full = L_FULL + pr * 4, fre = L_FREE + pr * 4, hdr = L_HDR + pr * 64;
  c.rs = mk_rsrc(c.wsme); c.ro = mk_out(p.out[rank]); c.ovoff = ((unsigned)lane >> 5) * (NN * 4u) + ((unsigned)lane & 31u) * 16u;
  if (wid < 2) {
    unsigned d[16], tk = sid, item = 0;
    if (lane < 2) lds_st(c, hdr + lane * 4, ((sid >> 2) * 8u + lane * 64u) * 4u + (sid & 3u));
    asm volatile("s_waitcnt vmcnt(0) lgkmcnt(0)" ::: "memory");
    if (lane == 0) lds_st(c, full, 2u);
    auto put = [&](int n) {
      i32x4 o[4]; unit_fix(item, d, o);
      for (const unsigned t0 = now(); (int)rfl(lds_ld(c, fre)) < n + 1 - RING;) {
        if (now() - t0 > DEADLINE) return false;
        __builtin_amdgcn_s_sleep(2);
      }
#pragma unroll
      for (int f = 0; f < 4; ++f) *(LAS i32x4*)(size_t)(ring + (n % RING) * 4096 + f * 1024) = o[f];
      if (lane == 0) lds_st(c, hdr + (n % RING) * 4, item * 4u + (tk & 3u));
      asm volatile("s_waitcnt lgkmcnt(0)" ::: "memory");
      if (lane == 0) lds_st(c, full, (unsigned)(n + 1));
      return true;
    };
    unsigned tb[5], nx = 0;
    if (lane == 0) nx = __hip_atomic_fetch_add((unsigned*)(c.wsme + WS_TKT), 1u, __ATOMIC_RELAXED, __HIP_MEMORY_SCOPE_AGENT);
#pragma unroll
    for (int i = 0; i < 5; ++i) tb[i] = ((const unsigned*)p.order)[i * 64 + lane];
    c.call = rfl(cnt) + 1u;
    const unsigned ord = L_ORD + pr * 2048;
#pragma unroll
    for (int i = 0; i < 5; ++i) lds_st(c, ord + (i * 64 + lane) * 4, tb[i]);
    const unsigned base = rfl(cnt) * (NALL - NSEND) - 2u * NSEND;
#pragma unroll 1
    for (int n = 2;; ++n) {
      tk = rfl(nx) - base;
      if (tk >= NALL) { if (lane == 0) lds_st(c, full, (unsigned)n | 0x10000u); break; }
      if (lane == 0) nx = __hip_atomic_fetch_add((unsigned*)(c.wsme + WS_TKT), 1u, __ATOMIC_RELAXED, __HIP_MEMORY_SCOPE_AGENT);
      if (tk < NUNIT) {
        item = rfl(*(volatile LAS unsigned short*)(size_t)(c.ldsb + ord + (tk >> 2) * 2u));
        item = item < (unsigned)NITEM ? item : 0u;
        unit_load(p, item, tk & 3u, lane, d);
      } else {
        if (item < (unsigned)NITEM) { wait_flags(c, WS_SHR + (lane & 31) * 4); if (rfl(lds_ld(c, L_BAD))) { if (lane == 0) lds_st(c, full, (unsigned)n | 0x20000u); break; } }
        item = tk >> 2;
#pragma unroll
        for (int f = 0; f < 4; ++f) {
          const auto v = __builtin_amdgcn_raw_buffer_load_b128(c.ro, c.ovoff, shr_off(tk - NUNIT, rank) + f * (2u * NN * 4u), 17);
          d[4 * f] = v[0]; d[4 * f + 1] = v[1]; d[4 * f + 2] = v[2]; d[4 * f + 3] = v[3];
        }
      }
      if (!put(n)) break;
    }
  } else {
    rsrc_t dst[7];
#pragma unroll
    for (int i = 0; i < 7; ++i) dst[i] = mk_rsrc(p.ws[(rank + 1u + (i + sid) % 7u) & 7u]);
    char* fbase = p.ws[(rank + 1u + (unsigned)(lane < 7 ? lane : 0)) & 7u];
    c.call = rfl(cnt) + 1u; c.stg = WS_STG + (c.call & 1u) * STG_BYTES;
    unsigned prev = 0, pm = 0, vo = c.voff, fs = 1024, fu = 0;
    bool outs = false;
#pragma unroll 1
    for (unsigned n = 0;; ++n) {
      for (const unsigned t0 = now(); ((fu = rfl(lds_ld(c, full))) & 0xffffu) <= n && fu < 0x10000u && now() - t0 < 2u * DEADLINE;) {
        if (pm != 0u) {
          asm volatile("s_waitcnt vmcnt(0)" ::: "memory");
          if (lane < 7 && ((pm >> lane) & 1u)) st_sys(fbase + prev, c.call);
          pm = 0;
        }
        __builtin_amdgcn_s_sleep(1);
      }
      if ((fu & 0xffffu) <= n) break;
      i32x4 o[4];
#pragma unroll
      for (int f = 0; f < 4; ++f) o[f] = *(const LAS i32x4*)(size_t)(ring + (n % RING) * 4096 + f * 1024);
      const unsigned u = rfl(lds_ld(c, hdr + (n % RING) * 4));
      unsigned so[7], m = 0x7fu;
      if (u >= NUNIT) {
        if (!outs) {
          outs = true; vo = c.ovoff; fs = 2u * NN * 4u;
#pragma unroll
          for (int k = 0; k < 7; ++k) dst[k] = mk_out(p.out[(rank + 1u + (k + sid) % 7u) & 7u]);
        }
        m = 0;
#pragma unroll
        for (int k = 0; k < 7; ++k) so[k] = shr_off(u - NUNIT, rank);
      } else {
        if (u >= 624u * 4u) m = (0x303u << (((u >> 5) - 78u) * 4u + (u & 1u) * 2u) >> (rank + 1u)) & 0x7fu;
#pragma unroll
        for (int k = 0; k < 7; ++k) {
          const unsigned kk = (k + sid) % 7u;
          so[k] = (m >> kk) & 1u ? c.stg + unit_off(u >> 5, 14u + 7u * ((u >> 2) & 7u) - kk, u & 3u) : OOB;
        }
      }
      asm volatile("s_waitcnt lgkmcnt(0)\n\ts_nop 4" : "+v"(o[0]), "+v"(o[1]), "+v"(o[2]), "+v"(o[3]) :: "memory");
      if (lane == 0) lds_st(c, fre, n + 1u);
#pragma unroll
      for (int f = 0; f < 4; ++f) {
#pragma unroll
        for (int k = 0; k < 7; ++k) st_wt(o[f], vo, dst[k], so[k] + f * fs);
        if (f == 0) asm volatile("s_waitcnt vmcnt(7)" ::: "memory"); else asm volatile("s_waitcnt vmcnt(%0)" :: "i"(PUSH_W) : "memory");
        if (f == 0) {
          if (lane < 7 && ((pm >> lane) & 1u)) st_sys(fbase + prev, c.call);
        }
      }
      prev = WS_FLG + rank * 16u + (u >> 2) * 128u + (u & 3u) * 4u; pm = m;
    }
    asm volatile("s_waitcnt vmcnt(0)" ::: "memory");
    if (lane < 7 && ((pm >> lane) & 1u)) st_sys(fbase + prev, c.call);
    if (lane < 7 && fu >> 16 == 1u) st_sys(fbase + WS_DON + rank * 128u + sid * 4u, c.call);
    if (sid == 0) {
      for (unsigned i = 0; i < 4; ++i) { const unsigned g = 2u * i + ((unsigned)lane >> 5); wait_flags(c, WS_DON + (g == rank ? g ^ 1u : g) * 128u + ((unsigned)lane & 31u) * 4u); }
      if (lds_ld(c, L_BAD)) __builtin_amdgcn_raw_buffer_store_b32(0x7fc00000u, c.ro, shr_off(0, rank), 0, 17);
    }
  }
}

__global__ void __launch_bounds__(256, 1) fused_gemm_gather(Params p) {
  extern __shared__ __attribute__((aligned(1024))) char smem[];
  const int tid = threadIdx.x, wid = __builtin_amdgcn_readfirstlane(tid >> 6), lane = tid & 63, wr = wid >> 1, wc = wid & 1;
  const int b = blockIdx.x;
  {
    unsigned long long pc; unsigned len;
    asm volatile("s_getpc_b64 %0\nfgg_code_begin:\n\ts_mov_b32 %1, fgg_code_end-fgg_code_begin" : "=s"(pc), "=s"(len));
    const rsrc_t rc = __builtin_amdgcn_make_buffer_rsrc((void*)pc, 0, len, 0x00020000);
    asm volatile("s_mov_b32 m0, %0\n\ts_nop 0\n\tbuffer_load_dword %1, %2, %3 offen lds"
                 :: "s"((unsigned)(size_t)(LAS char*)smem + (96u << 10) + wid * 256), "v"(lane * 64), "s"(rc), "s"((((b >> 3) * 4 + wid) & 15) * 4096) : "memory");
  }
  if (b < 16 && wid < 2) {
    const unsigned sid = (unsigned)(b * 2 + wid);
    const rsrc_t rx = __builtin_amdgcn_make_buffer_rsrc((void*)p.x, 0, (unsigned)MM * KS, 0x00020000);
    const unsigned vo = (unsigned)(lane & 15) * KS + (unsigned)(lane >> 4) * 16u, so = ((sid >> 2) * 256u + (sid & 3u) * 64u) * KS;
#pragma unroll
    for (int i = 0; i < 8; ++i)
      asm volatile("s_mov_b32 m0, %0\n\ts_nop 0\n\tbuffer_load_dwordx4 %1, %2, %3 offen lds"
                   :: "s"((unsigned)(size_t)(LAS char*)smem + wid * (RING * 4096) + i * 1024), "v"(vo), "s"(rx), "s"(so + ((i >> 2) * 2048 + (i & 3) * 16) * KS) : "memory");
  }
  const unsigned rank = (unsigned)p.rank & 7u;
  Ctx c;
  c.lane = lane; c.wid = wid; c.wsme = p.ws[rank];
  const unsigned cnt = ld_sys(c.wsme + WS_CNT + b * 128);
  c.ldsb = (unsigned)(size_t)(LAS char*)smem;
  c.voff = lane * 16;
  if (tid < 16) lds_st(c, L_RDY + tid * 4, 0u);
  asm volatile("s_waitcnt lgkmcnt(0)\n\ts_barrier" ::: "memory");
  if (__builtin_expect(b < 16, 1)) {
    if (b == 0 && tid == 128) st_sys(c.wsme + WS_ROT + (__builtin_amdgcn_s_getreg(20 | (3 << 11)) & 1u) * 4u, cnt + 1u);
    pusher(p, c, b, wid, rank, cnt);
    __syncthreads();
    if (tid == 0) st_sys(c.wsme + WS_CNT + b * 128, cnt + 1u);
  } else {
    c.aoff = c.ldsb + c.voff + wr * 4096; c.boff = c.ldsb + c.voff + wc * 4096; c.aoffH = c.aoff + 65536; c.boffH = c.boff + 65536;
    asm volatile("" : "+v"(c.aoff), "+v"(c.boff), "+v"(c.aoffH), "+v"(c.boffH));
    c.m0base = c.ldsb + wid * 4096;
    c.rs = mk_rsrc(c.wsme);
    c.ro = mk_out(p.out[rank]);
    c.ovoff = (unsigned)(wr * 128 + 4 * (lane >> 4)) * (NN * 4u) + (unsigned)(wc * 64 + 4 * (lane & 15)) * 4u;
    const unsigned scale = rfl(__float_as_uint(p.sx[0] * p.sw[0]));
    {
      const unsigned g = (unsigned)(b - 16);
      unsigned d[3][16];
#pragma unroll
      for (int i = 0; i < 3; ++i) { const unsigned it = g + 240u * i; unit_load(p, it < (unsigned)NITEM ? it : g, (unsigned)wid, lane, d[i]); }
      c.call = rfl(cnt) + 1u; c.stg = WS_STG + (c.call & 1u) * STG_BYTES;
#pragma unroll
      for (int i = 0; i < 3; ++i) {
        const unsigned it = g + 240u * i;
        if (it < (unsigned)NITEM) {
          i32x4 o[4]; unit_fix(it, d[i], o);
          asm volatile("s_nop 4" ::: "memory");
#pragma unroll
          for (int f = 0; f < 4; ++f) st_wt(o[f], c.voff, c.rs, c.stg + unit_off(it >> 3, it & 7u, (unsigned)wid) + f * 1024);
        }
      }
      asm volatile("s_waitcnt vmcnt(0)" ::: "memory");
      const unsigned it = g + 240u * (unsigned)lane;
      if (lane < 3 && it < (unsigned)NITEM) st_sys(c.wsme + WS_FLG + it * 128u + rank * 16u + wid * 4, c.call);
    }
    unsigned rot;
    {
      const unsigned t0 = now();
      unsigned long long m;
      while ((m = __builtin_amdgcn_ballot_w64(ld_sys(c.wsme + WS_ROT + (lane & 1) * 4) == c.call)) == 0ull) {
        if (now() - t0 > DEADLINE) {
          if (lane == 0) { __hip_atomic_store((unsigned*)(c.wsme + WS_TMO), c.call, __ATOMIC_RELAXED, __HIP_MEMORY_SCOPE_AGENT); lds_st(c, L_BAD, 0x7fc00000u); }
          break;
        }
        __builtin_amdgcn_s_sleep(2);
      }
      rot = (m & 1ull) ? 0u : 1u;
    }
    const int xc = (int)((b + rot) & 7u), u60 = (xc & 1) * 30 + (b >> 3) - 2, ti = (xc >> 1) * 4 + (u60 & 3), q = u60 >> 2;
    body(c, scale, ti, q, rank);
  }
}

static void make_order(unsigned short* order) {
  constexpr double TA = 1.25, PF = 2.0, PE = 2.0, PO = 0.0, SO = 0.1;
  struct It { double key; int item; };
  It v[NITEM]; int n = 0;
  for (int ks = 0; ks < 8; ++ks) {
    for (int i = 0; i < 16; ++i) v[n++] = {(ks - 7) * TA - 0.2 + i * 1e-3, i * 8 + ks};
    for (int e = 0; e < 4; ++e) v[n++] = {(ks - 7) * TA - (ks ? 0.3 : 0.1) + e * 1e-3, 128 + (60 + e) * 8 + ks};
  }
  for (int q = 0; q < 15; ++q)
    for (int m = 0; m < 32; ++m) {
      const double ph = q < 3 ? PF : q < 7 ? PE : PO + SO * (14 - q) / 7.0;
      v[n++] = {m + ph + q * 1e-3, 128 + jmap(q, 1 + (m >> 3)) * 8 + (m & 7)};
    }
  std::stable_sort(v, v + n, [](const It& a, const It& b) { return a.key < b.key; });
  for (int i = 0; i < NITEM; ++i) order[i] = (unsigned short)v[i].item;
}

extern "C" void kernel_launch(int rank, int nranks, ncclComm_t comm, hipStream_t stream,
                              void* const* d_in, const int* in_sizes, int n_in,
                              void* d_out, int out_size, const int* in_dtypes, int out_dtype,
                              void* const* d_out_all, void* const* d_ws_all, size_t ws_bytes) {
  (void)comm; (void)in_dtypes; (void)out_dtype; (void)d_out;
  if (nranks != NR || n_in != 4 || (unsigned)rank >= (unsigned)NR || ws_bytes < (size_t)WS_USED || in_sizes[0] != MM * KS || in_sizes[1] != KS * NN || out_size != MM * NN) {
    fprintf(stderr, "kernel_launch: unexpected problem shape\n"); return;
  }
  HIP_CHECK(hipFuncSetAttribute((const void*)fused_gemm_gather, hipFuncAttributeMaxDynamicSharedMemorySize, (int)LDS_BYTES));
  Params p;
  p.x = (const unsigned char*)d_in[0]; p.w = (const unsigned char*)d_in[1]; p.sx = (const float*)d_in[2]; p.sw = (const float*)d_in[3];
  p.rank = rank; p.pad = 0;
  for (int i = 0; i < NR; ++i) { p.ws[i] = (char*)d_ws_all[i]; p.out[i] = (float*)d_out_all[i]; }
  struct Order { unsigned short v[NITEM]; Order() { make_order(v); } };
  static const Order ord;
  std::copy(ord.v, ord.v + NITEM, p.order);
  fused_gemm_gather<<<dim3(256), dim3(256), LDS_BYTES, stream>>>(p);
  HIP_CHECK(hipGetLastError());
}
```
